```python
import numpy as np

kernel_name = "gpu_a2a_gemm_m4096_k4096_n16384_bf16_q8"


ABSMAX_RTOL = 0.02

COMPUTE_DTYPE = "bfloat16"
EPILOGUE = "quant_int8"
IN_QUANT = None
OUT_QUANT = "int8"
IN_DTYPES = {"x": "bfloat16", "w_mat": "bfloat16"}
WORLD_SIZE = 8

def _to_bf16(x):
    u = np.ascontiguousarray(x, dtype=np.float32).view(np.uint32)
    return ((u + 0x7FFF + ((u >> 16) & 1)) & 0xFFFF0000).view(np.float32)

def setup_inputs():

    x = _to_bf16(np.random.randn(4096, 4096).astype(np.float32))
    w_mat = _to_bf16((np.random.randn(4096, 16384) / np.sqrt(4096)).astype(np.float32))
    return {"x": x, "w_mat": w_mat}

def _quant_dequant(y, amax):
    scale = amax / 127.0
    q = np.clip(np.round(y / scale), -127.0, 127.0)
    return (q * scale).astype(np.float32)
def _epilogue(y):
    return _quant_dequant(y, np.abs(y).max())

def reference(x, w_mat):
    y = (x @ w_mat).astype(np.float32)
    return _epilogue(y)
```

```cpp
#include <hip/hip_runtime.h>
#include <rccl/rccl.h>
#include <cstdio>
#include <type_traits>

#define HIP_CHECK(expr) do { hipError_t _e = (expr); if (_e != hipSuccess) { \
    fprintf(stderr, "HIP error %s at %s:%d\n", hipGetErrorString(_e), __FILE__, __LINE__); \
    return; } } while (0)

#define LAS __attribute__((address_space(3)))
#define CAS __attribute__((address_space(4)))
typedef short bf16x8 __attribute__((ext_vector_type(8)));
typedef short bf16x4 __attribute__((ext_vector_type(4)));
typedef float f32x4 __attribute__((ext_vector_type(4)));
typedef unsigned u32x4 __attribute__((ext_vector_type(4)));
typedef unsigned long long u64;

namespace {

constexpr int NRANK = 8;
constexpr int NN = 16384, KK = 4096;
constexpr int NT = KK / 64;
constexpr int A_ST = 32768, B_ST = 16384, NA = 3, NB = 4;
constexpr int B_OFF = 0, A_OFF = NB * B_ST, LDS_BYTES = NA * A_ST + NB * B_ST;
constexpr int NTHREADS = 768, NWG = 256;

constexpr size_t WS_SEQ = 0, WS_TMO = 128;
constexpr size_t WS_FLAGS = 4096;
constexpr size_t WS_LGRAN = 8192;
constexpr size_t WS_GGRAN = 16384;
constexpr int NREP = 32;
constexpr size_t WS_STAGE = 1u << 20;
constexpr size_t WS_NEED = WS_STAGE + (size_t)56 * 65536;

constexpr u64 DEADLINE_TICKS = 100000000ull;
#define PACE 6
#define PUB_DELAY 6

struct Params {
  const char* x;
  const char* w;
  signed char* out;
  char* ws[NRANK];
  int rank;
};

#define WAIT_V(n) asm volatile("s_waitcnt vmcnt(" #n ")" ::: "memory")
#define WAIT_L(n) asm volatile("s_waitcnt lgkmcnt(" #n ")" ::: "memory")
#define BAR __builtin_amdgcn_s_barrier()
#define SCHED __builtin_amdgcn_sched_barrier(0)
#define IC(x) std::integral_constant<int, (x)>{}

__device__ __forceinline__ void dma4(unsigned lds_addr, const char* gbase, unsigned v0, unsigned v1, unsigned v2, unsigned v3) {
  asm volatile("s_mov_b32 m0, %0\n\ts_nop 0\n\t"
               "global_load_lds_dwordx4 %2, %1\n\t"
               "global_load_lds_dwordx4 %3, %1 offset:1024\n\t"
               "global_load_lds_dwordx4 %4, %1 offset:2048\n\t"
               "global_load_lds_dwordx4 %5, %1 offset:3072"
               :: "s"(lds_addr), "s"(gbase), "v"(v0), "v"(v1), "v"(v2), "v"(v3) : "memory");
}

__device__ __forceinline__ unsigned sload_u32(const char* p) { return *(const CAS unsigned*)p; }
__device__ __forceinline__ u64 now() { return __builtin_amdgcn_s_memrealtime(); }
__device__ __forceinline__ bool timed_out(unsigned* tmo, unsigned seq) {
  return (unsigned)__builtin_amdgcn_readfirstlane(__hip_atomic_load(tmo, __ATOMIC_RELAXED, __HIP_MEMORY_SCOPE_AGENT)) == seq;
}
__device__ __forceinline__ void set_timeout(unsigned* tmo, unsigned seq) { __hip_atomic_store(tmo, seq, __ATOMIC_RELAXED, __HIP_MEMORY_SCOPE_AGENT); }
__device__ __forceinline__ void acquire_sys() { asm volatile("buffer_inv sc0 sc1" ::: "memory"); }

__device__ __forceinline__ unsigned wave_max_u32(unsigned v) {
  v = max(v, (unsigned)__builtin_amdgcn_update_dpp(0, (int)v, 0x111, 0xf, 0xf, true));
  v = max(v, (unsigned)__builtin_amdgcn_update_dpp(0, (int)v, 0x112, 0xf, 0xf, true));
  v = max(v, (unsigned)__builtin_amdgcn_update_dpp(0, (int)v, 0x114, 0xf, 0xf, true));
  v = max(v, (unsigned)__builtin_amdgcn_update_dpp(0, (int)v, 0x118, 0xf, 0xf, true));
  v = max(v, (unsigned)__builtin_amdgcn_update_dpp(0, (int)v, 0x142, 0xa, 0xf, true));
  v = max(v, (unsigned)__builtin_amdgcn_update_dpp(0, (int)v, 0x143, 0xc, 0xf, true));
  return (unsigned)__builtin_amdgcn_readlane((int)v, 63);
}

__device__ __forceinline__ int step_k(int t, int rank) {
  if (t < 8) return rank * 512 + t * 64;
  const int u = t - 8, j = (u * 37) >> 8, i = u - j * 7;
  return ((rank + 1 + i) & 7) * 512 + j * 64;
}

__global__ __launch_bounds__(NTHREADS) void a2a_gemm_q8(Params P) {
  extern __shared__ __attribute__((aligned(16))) unsigned char shm[];
  LAS unsigned char* lds = (LAS unsigned char*)shm;
  const unsigned ldsb = (unsigned)(size_t)lds;
  const int tid = threadIdx.x, wid = __builtin_amdgcn_readfirstlane(tid >> 6), lane = tid & 63;
  const int b = blockIdx.x, xcd = b & 7, idx = b >> 3;
  const int ct = xcd * 16 + (idx >> 1), rt = idx & 1;
  const int rank = P.rank;
  char* const myws = P.ws[rank];
  unsigned* const tmo = (unsigned*)(myws + WS_TMO);

  if (wid >= 8) {
    if (wid < 10) {
      const int a = wid - 8;
      unsigned vS[2], vL[4];
#pragma unroll
      for (int h = 0; h < 2; ++h) { const int f = (4 * h + (lane >> 4)) & 7; vS[h] = (unsigned)((lane >> 3) * 128 + (((lane & 7) ^ f) * 16)); }
#pragma unroll
      for (int i = 0; i < 4; ++i) { const int f = (4 * i + (lane >> 4)) & 7; vL[i] = (unsigned)((lane >> 3) * 1024 + (((lane & 7) ^ f) * 16) + i * (8192 - 1024)); }
      const char* xb = P.x + (size_t)(rank * 512 + rt * 256 + a * 128) * 1024;
      const char* sbp = myws + WS_STAGE + (size_t)rt * 32768 + a * 16384;
      const char* flags = myws + WS_FLAGS;
      const unsigned ldst = ldsb + A_OFF + a * 16384;
      const unsigned foff = (unsigned)(lane & 7) * 16;
      auto issue_half = [&](int t, int h) {
        const unsigned l = ldst + (t % NA) * A_ST + h * 8192;
        if (t < 8) { const char* g = xb + t * 128 + h * (64 * 1024);
          dma4(l, g, vL[0], vL[1], vL[2], vL[3]); dma4(l + 4096, g + 32 * 1024, vL[0], vL[1], vL[2], vL[3]); }
        else { const char* g = sbp + (size_t)(t - 8) * 65536 + h * 8192;
          dma4(l, g, vS[0], vS[1], vS[0], vS[1]); dma4(l + 4096, g + 4096, vS[0], vS[1], vS[0], vS[1]); }
      };
      issue_half(0, 0); issue_half(0, 1); issue_half(1, 0); issue_half(1, 1);
      const unsigned seq = sload_u32(myws + WS_SEQ) + 1u;
      WAIT_V(16); BAR;
      int cur_j = -1; unsigned mask = 0; bool dead = false;
      for (int t = 0; t < NT; ++t) {
        const int s = t + 2;
        if (s >= 8 && s < NT && !dead) {
          const int u = s - 8, j = (u * 37) >> 8, src = (rank + 1 + (u - j * 7)) & 7;
          if (j != cur_j || !((mask >> src) & 1u)) {
            cur_j = j;
            const u64 t0 = now();
            for (;;) {
              u32x4 f;
              asm volatile("global_load_dwordx4 %0, %1, %2 sc0 sc1\n\ts_waitcnt vmcnt(0)" : "=&v"(f) : "v"(foff), "s"(flags + j * 128) : "memory");
              const bool ok = f[0] >= seq && f[1] >= seq && f[2] >= seq && f[3] >= seq;
              mask = (unsigned)__builtin_amdgcn_ballot_w64(ok) & 0xffu;
              if ((mask >> src) & 1u) break;
              if (timed_out(tmo, seq) || now() - t0 > DEADLINE_TICKS) { set_timeout(tmo, seq); dead = true; break; }
              __builtin_amdgcn_s_sleep(1);
            }
            acquire_sys();
          }
        }
        bool poll = false; int j1 = 0;
        if (s + 1 >= 8 && s + 1 < NT && !dead) {
          const int u = s + 1 - 8; j1 = (u * 37) >> 8; const int src = (rank + 1 + (u - j1 * 7)) & 7;
          poll = (j1 != cur_j) || !((mask >> src) & 1u);
        }
        const bool live = s < NT && !dead;
        if (poll) {
          u32x4 f;
          asm volatile("global_load_dwordx4 %0, %1, %2 sc0 sc1" : "=&v"(f) : "v"(foff), "s"(flags + j1 * 128) : "memory");
          issue_half(s, 0);
          BAR;
          issue_half(s, 1);
          WAIT_V(16);
          asm volatile("" : "+v"(f));
          const bool ok = f[0] >= seq && f[1] >= seq && f[2] >= seq && f[3] >= seq;
          const unsigned nm = (unsigned)__builtin_amdgcn_ballot_w64(ok) & 0xffu;
          cur_j = j1; mask = nm;
          if (nm) acquire_sys();
          BAR;
        } else if (live) {
          issue_half(s, 0);
          BAR;
          issue_half(s, 1);
          WAIT_V(16);
          BAR;
        } else {
          BAR;
          WAIT_V(0);
          BAR;
        }
      }
      BAR;
    } else if (wid == 10) {
      unsigned vB[4];
#pragma unroll
      for (int i = 0; i < 4; ++i) { const int q = lane >> 4, s = (q << 1) | ((i >> 1) & 1), u = ((lane & 15) >> 1) ^ s;
        vB[i] = (unsigned)(q * (NN * 2) + (2 * u + (lane & 1)) * 16 + i * (4 * NN * 2 - 1024)); }
      const char* wb = P.w + (size_t)ct * 256;
      const unsigned ldst = ldsb + B_OFF;
      auto issue_half = [&](int t, int h) {
        const char* g = wb + (size_t)(step_k(t, rank) + h * 32) * (NN * 2);
        const unsigned l = ldst + (t % NB) * B_ST + h * 8192;
        dma4(l, g, vB[0], vB[1], vB[2], vB[3]); dma4(l + 4096, g + (size_t)16 * NN * 2, vB[0], vB[1], vB[2], vB[3]);
      };
      issue_half(0, 0); issue_half(0, 1); issue_half(1, 0); issue_half(1, 1); issue_half(2, 0); issue_half(2, 1);
      WAIT_V(32); BAR;
      for (int t = 0; t < NT - 3; ++t) {
        issue_half(t + 3, 0);
        BAR;
        issue_half(t + 3, 1);
        WAIT_V(32);
        BAR;
      }
      BAR; WAIT_V(16); BAR;
      BAR; WAIT_V(0); BAR;
      BAR; BAR;
      BAR;
    } else {
      if (b >= 224) return;
      const int j = b / 28, r = b - j * 28, i = r >> 2, q = r & 3;
      const int dst = (rank + 1 + i) & 7, u = 7 * j + (6 - i);
      const char* src = P.x + (size_t)(dst * 512 + q * 128) * 1024 + j * 128;
      char* dws = P.ws[dst];
      char* dbase = dws + WS_STAGE + (size_t)u * 65536 + q * 16384;
      const unsigned voff = (unsigned)((lane >> 3) * 1024 + (lane & 7) * 16);
      u32x4 d[16];
#pragma unroll
      for (int k = 0; k < 16; ++k)
        asm volatile("global_load_dwordx4 %0, %1, %2" : "=&v"(d[k]) : "v"(voff), "s"(src + k * 8192) : "memory");
      const unsigned seq = sload_u32(myws + WS_SEQ) + 1u;
      const int start = PACE * j;
      for (int s = 0; s < start; ++s) BAR;
      WAIT_V(0);
      const unsigned soff = (unsigned)lane * 16;
#pragma unroll
      for (int k = 0; k < 16; ++k)
        asm volatile("global_store_dwordx4 %0, %1, %2 sc0 sc1\n\ts_nop 1" :: "v"(soff), "v"(d[k]), "s"(dbase + k * 1024) : "memory");
      for (int s = 0; s < PUB_DELAY; ++s) BAR;
      WAIT_V(0);
      if (lane == 0)
        __hip_atomic_store((unsigned*)(dws + WS_FLAGS + j * 128 + rank * 16 + q * 4), seq, __ATOMIC_RELAXED, __HIP_MEMORY_SCOPE_SYSTEM);
    }
    return;
  }

  const int grp = wid >> 2, wm = wid & 3, wn = grp;
  const int fr = lane & 15, fq = lane >> 4;
  unsigned aoff[2];
#pragma unroll
  for (int kh = 0; kh < 2; ++kh) aoff[kh] = ldsb + (unsigned)(A_OFF + (wm * 64 + fr) * 128 + (((4 * kh + fq) ^ ((fr >> 1) & 7)) * 16));
  unsigned boff[4];
  { const int q = (lane & 15) >> 2, p = lane & 3; const int s = (q << 1) | (fq & 1);
#pragma unroll
    for (int n = 0; n < 4; ++n) { const int u = (wn * 4 + n) ^ s; boff[n] = ldsb + (unsigned)(B_OFF + (8 * fq + q) * 256 + u * 32 + p * 8); } }
  unsigned aoffH[2] = {aoff[0] + 65536u, aoff[1] + 65536u};
  asm volatile("" : "+v"(aoffH[0]), "+v"(aoffH[1]));

  f32x4 acc[4][4];
#pragma unroll
  for (int m = 0; m < 4; ++m)
#pragma unroll
    for (int n = 0; n < 4; ++n) acc[m][n] = (f32x4){0.f, 0.f, 0.f, 0.f};
  bf16x8 af[4][2], bfr[4][2];
  const unsigned seq = sload_u32(myws + WS_SEQ) + 1u;

  BAR;
  if (grp == 1) BAR;

#define TRD(dst, addr, off) asm volatile("ds_read_b64_tr_b16 %0, %1 offset:%2" : "=&v"(dst) : "v"(addr), "i"(off) : "memory")
#define ARD(dst, addr, off) asm volatile("ds_read_b128 %0, %1 offset:%2" : "=&v"(dst) : "v"(addr), "i"(off) : "memory")
  auto step = [&](auto SAc, auto SBc) {
    constexpr int SA = decltype(SAc)::value, SB = decltype(SBc)::value;
    bf16x4 lo[4][2], hi[4][2];
#pragma unroll
    for (int n = 0; n < 4; ++n) {
      TRD(lo[n][0], boff[n], SB * B_ST); TRD(hi[n][0], boff[n], SB * B_ST + 1024);
      TRD(lo[n][1], boff[n], SB * B_ST + 8192); TRD(hi[n][1], boff[n], SB * B_ST + 9216);
    }
#pragma unroll
    for (int m = 0; m < 4; ++m) {
      if constexpr (SA < 2) { ARD(af[m][0], aoff[0], SA * A_ST + m * 2048); ARD(af[m][1], aoff[1], SA * A_ST + m * 2048); }
      else { ARD(af[m][0], aoffH[0], m * 2048); ARD(af[m][1], aoffH[1], m * 2048); }
    }
    WAIT_L(0);
    SCHED;
#pragma unroll
    for (int n = 0; n < 4; ++n)
#pragma unroll
      for (int kh = 0; kh < 2; ++kh) bfr[n][kh] = __builtin_shufflevector(lo[n][kh], hi[n][kh], 0, 1, 2, 3, 4, 5, 6, 7);
    BAR;
    __builtin_amdgcn_s_setprio(1);
#pragma unroll
    for (int kh = 0; kh < 2; ++kh)
#pragma unroll
      for (int m = 0; m < 4; ++m)
#pragma unroll
        for (int n = 0; n < 4; ++n) acc[m][n] = __builtin_amdgcn_mfma_f32_16x16x32_bf16(bfr[n][kh], af[m][kh], acc[m][n], 0, 0, 0);
    __builtin_amdgcn_s_setprio(0);
    BAR;
    SCHED;
  };
  for (int it = 0; it < 5; ++it) {
    step(IC(0), IC(0)); step(IC(1), IC(1)); step(IC(2), IC(2)); step(IC(0), IC(3));
    step(IC(1), IC(0)); step(IC(2), IC(1)); step(IC(0), IC(2)); step(IC(1), IC(3));
    step(IC(2), IC(0)); step(IC(0), IC(1)); step(IC(1), IC(2)); step(IC(2), IC(3));
  }
  step(IC(0), IC(0)); step(IC(1), IC(1)); step(IC(2), IC(2)); step(IC(0), IC(3));

  float mx = 0.f;
#pragma unroll
  for (int m = 0; m < 4; ++m)
#pragma unroll
    for (int n = 0; n < 4; ++n) {
      mx = __builtin_fmaxf(mx, __builtin_fmaxf(__builtin_fabsf(acc[m][n][0]), __builtin_fabsf(acc[m][n][1])));
      mx = __builtin_fmaxf(mx, __builtin_fmaxf(__builtin_fabsf(acc[m][n][2]), __builtin_fabsf(acc[m][n][3])));
    }
  const unsigned wmx = wave_max_u32(__float_as_uint(mx));
  if (grp == 0) BAR;
  LAS unsigned* red = (LAS unsigned*)lds;
  if (lane == 0) red[wid] = wmx;
  WAIT_L(0); BAR;

  if (wid == 0) {
    unsigned v = red[lane & 7];
    v = max(v, (unsigned)__builtin_amdgcn_update_dpp(0, (int)v, 0x111, 0xf, 0xf, true));
    v = max(v, (unsigned)__builtin_amdgcn_update_dpp(0, (int)v, 0x112, 0xf, 0xf, true));
    v = max(v, (unsigned)__builtin_amdgcn_update_dpp(0, (int)v, 0x114, 0xf, 0xf, true));
    const unsigned gmx = (unsigned)__builtin_amdgcn_readlane((int)v, 7);
    u64* lg = (u64*)(myws + WS_LGRAN);
    if (lane == 0) __hip_atomic_store(lg + b, ((u64)seq << 32) | gmx, __ATOMIC_RELAXED, __HIP_MEMORY_SCOPE_AGENT);
    bool dead = false;
    if (b < NRANK) {
      const u64 t0 = now(); unsigned am = 0;
      for (;;) {
        bool ok = true; am = 0;
#pragma unroll
        for (int k = 0; k < 4; ++k) { const u64 g = __hip_atomic_load(lg + k * 64 + lane, __ATOMIC_RELAXED, __HIP_MEMORY_SCOPE_AGENT);
          ok = ok && (unsigned)(g >> 32) >= seq; am = max(am, (unsigned)g); }
        if (__builtin_amdgcn_ballot_w64(ok) == ~0ull) break;
        if (timed_out(tmo, seq) || now() - t0 > DEADLINE_TICKS) { set_timeout(tmo, seq); dead = true; break; }
        __builtin_amdgcn_s_sleep(1);
      }
      am = wave_max_u32(am);
      if (!dead && lane < NREP)
        __hip_atomic_store((u64*)(P.ws[b] + WS_GGRAN + lane * 128 + rank * 8), ((u64)seq << 32) | am, __ATOMIC_RELAXED, __HIP_MEMORY_SCOPE_SYSTEM);
    }
    const u64* gg = (const u64*)(myws + WS_GGRAN + (b & (NREP - 1)) * 128) + (lane & 7);
    const u64 t0 = now(); unsigned am = 0;
    while (!dead) {
      const u64 g = __hip_atomic_load(gg, __ATOMIC_RELAXED, __HIP_MEMORY_SCOPE_SYSTEM);
      am = (unsigned)g;
      if (__builtin_amdgcn_ballot_w64((unsigned)(g >> 32) >= seq) == ~0ull) break;
      if (timed_out(tmo, seq) || now() - t0 > DEADLINE_TICKS) { set_timeout(tmo, seq); dead = true; break; }
      __builtin_amdgcn_s_sleep(1);
    }
    am = max(am, (unsigned)__builtin_amdgcn_update_dpp(0, (int)am, 0x111, 0xf, 0xf, true));
    am = max(am, (unsigned)__builtin_amdgcn_update_dpp(0, (int)am, 0x112, 0xf, 0xf, true));
    am = max(am, (unsigned)__builtin_amdgcn_update_dpp(0, (int)am, 0x114, 0xf, 0xf, true));
    am = (unsigned)__builtin_amdgcn_readlane((int)am, 7);
    const bool bad = dead || timed_out(tmo, seq);
    if (lane == 0) { red[8] = am; red[9] = bad ? 1u : 0u;
      if (b == 0) __hip_atomic_store((unsigned*)(myws + WS_SEQ), seq, __ATOMIC_RELAXED, __HIP_MEMORY_SCOPE_AGENT); }
  }
  WAIT_L(0); BAR;
  const float amax = __uint_as_float(red[8]);
  const bool bad = red[9] != 0u;
  const float inv = amax > 0.f ? 1.0f / (amax / 127.0f) : 0.f;

  signed char* orow = P.out + (size_t)(rt * 256 + wm * 64 + fr) * NN + ct * 128 + wn * 64 + fq * 16;
#pragma unroll
  for (int m = 0; m < 4; ++m) {
    unsigned d[4];
#pragma unroll
    for (int n = 0; n < 4; ++n) {
      unsigned q[4];
#pragma unroll
      for (int e = 0; e < 4; ++e) q[e] = __float_as_uint(__builtin_fmaf(acc[m][n][e], inv, 12582912.0f));
      const unsigned p01 = __builtin_amdgcn_perm(q[1], q[0], 0x0c0c0400u), p23 = __builtin_amdgcn_perm(q[3], q[2], 0x04000c0cu);
      d[n] = p01 | p23;
    }
    auto r0 = __builtin_amdgcn_permlane32_swap(d[0], d[2], false, false); d[0] = r0[0]; d[2] = r0[1];
    auto r1 = __builtin_amdgcn_permlane32_swap(d[1], d[3], false, false); d[1] = r1[0]; d[3] = r1[1];
    auto r2 = __builtin_amdgcn_permlane16_swap(d[0], d[1], false, false); d[0] = r2[0]; d[1] = r2[1];
    auto r3 = __builtin_amdgcn_permlane16_swap(d[2], d[3], false, false); d[2] = r3[0]; d[3] = r3[1];
    u32x4 o = (u32x4){d[0], d[1], d[2], d[3]};
    if (bad) o = (u32x4){0x80808080u, 0x80808080u, 0x80808080u, 0x80808080u};
    *(u32x4*)(orow + (size_t)m * 16 * NN) = o;
  }
}

}

extern "C" void kernel_launch(int rank, int nranks, ncclComm_t comm,
                              hipStream_t stream,
                              void* const* d_in, const int* in_sizes, int n_in,
                              void* d_out, int out_size,
                              const int* in_dtypes, int out_dtype,
                              void* const* d_out_all,
                              void* const* d_ws_all, size_t ws_bytes) {
  (void)comm; (void)in_dtypes; (void)out_dtype; (void)d_out_all;
  if (nranks != NRANK || (unsigned)rank >= (unsigned)NRANK || n_in != 2 || ws_bytes < WS_NEED ||
      in_sizes[0] != 4096 * 512 || in_sizes[1] != KK * NN || out_size != 512 * NN) {
    fprintf(stderr, "kernel_launch: unexpected problem shape\n");
    return;
  }
  static bool init = false;
  if (!init) {
    HIP_CHECK(hipFuncSetAttribute((const void*)a2a_gemm_q8, hipFuncAttributeMaxDynamicSharedMemorySize, LDS_BYTES));
    init = true;
  }
  Params P;
  P.x = (const char*)d_in[0];
  P.w = (const char*)d_in[1];
  P.out = (signed char*)d_out;
  for (int p = 0; p < NRANK; ++p) P.ws[p] = (char*)d_ws_all[p];
  P.rank = rank;
  hipLaunchKernelGGL(a2a_gemm_q8, dim3(NWG), dim3(NTHREADS), LDS_BYTES, stream, P);
  HIP_CHECK(hipGetLastError());
}
```

```cpp
#include <hip/hip_runtime.h>
#include <rccl/rccl.h>
#include <cstdio>
#include <type_traits>

#define HIP_CHECK(expr) do { hipError_t _e = (expr); if (_e != hipSuccess) { \
    fprintf(stderr, "HIP error %s at %s:%d\n", hipGetErrorString(_e), __FILE__, __LINE__); \
    return; } } while (0)

#define LAS __attribute__((address_space(3)))
#define CAS __attribute__((address_space(4)))
typedef short bf16x8 __attribute__((ext_vector_type(8)));
typedef short bf16x4 __attribute__((ext_vector_type(4)));
typedef float f32x4 __attribute__((ext_vector_type(4)));
typedef unsigned u32x4 __attribute__((ext_vector_type(4)));
typedef unsigned long long u64;

namespace {

constexpr int NRANK = 8;
constexpr int NN = 16384, KK = 4096;
constexpr int NT = KK / 64;
constexpr int A_ST = 32768, B_ST = 16384, NA = 3, NB = 4;
constexpr int B_OFF = 0, A_OFF = NB * B_ST, LDS_BYTES = NA * A_ST + NB * B_ST;
constexpr int NTHREADS = 768, NWG = 256;

constexpr size_t WS_SEQ = 0, WS_TMO = 128;
constexpr size_t WS_FLAGS = 4096;
constexpr size_t WS_LGRAN = 8192;
constexpr size_t WS_GGRAN = 16384;
constexpr int NREP = 32;
constexpr size_t WS_STAGE = 1u << 20;
constexpr size_t WS_NEED = WS_STAGE + (size_t)56 * 65536;

constexpr u64 DEADLINE_TICKS = 100000000ull;
#define PACE 6
#define PUB_DELAY 6

struct Params {
  const char* x;
  const char* w;
  signed char* out;
  char* ws[NRANK];
  int rank;
};

#define WAIT_V(n) asm volatile("s_waitcnt vmcnt(" #n ")" ::: "memory")
#define WAIT_L(n) asm volatile("s_waitcnt lgkmcnt(" #n ")" ::: "memory")
#define BAR __builtin_amdgcn_s_barrier()
#define SCHED __builtin_amdgcn_sched_barrier(0)
#define IC(x) std::integral_constant<int, (x)>{}

#define DEF_DMA4(NAME, POL) \
__device__ __forceinline__ void NAME(unsigned lds_addr, const char* gbase, unsigned v0, unsigned v1, unsigned v2, unsigned v3) { \
  asm volatile("s_mov_b32 m0, %0\n\ts_nop 0\n\t" \
               "global_load_lds_dwordx4 %2, %1" POL "\n\t" \
               "global_load_lds_dwordx4 %3, %1 offset:1024" POL "\n\t" \
               "global_load_lds_dwordx4 %4, %1 offset:2048" POL "\n\t" \
               "global_load_lds_dwordx4 %5, %1 offset:3072" POL \
               :: "s"(lds_addr), "s"(gbase), "v"(v0), "v"(v1), "v"(v2), "v"(v3) : "memory"); \
}
DEF_DMA4(dma4, "")
DEF_DMA4(dma4_sys, " sc0 sc1")

__device__ __forceinline__ unsigned sload_u32(const char* p) { return *(const CAS unsigned*)p; }
__device__ __forceinline__ u64 now() { return __builtin_amdgcn_s_memrealtime(); }
__device__ __forceinline__ bool timed_out(unsigned* tmo, unsigned seq) {
  return (unsigned)__builtin_amdgcn_readfirstlane(__hip_atomic_load(tmo, __ATOMIC_RELAXED, __HIP_MEMORY_SCOPE_AGENT)) == seq;
}
__device__ __forceinline__ void set_timeout(unsigned* tmo, unsigned seq) { __hip_atomic_store(tmo, seq, __ATOMIC_RELAXED, __HIP_MEMORY_SCOPE_AGENT); }

__device__ __forceinline__ unsigned wave_max_u32(unsigned v) {
  v = max(v, (unsigned)__builtin_amdgcn_update_dpp(0, (int)v, 0x111, 0xf, 0xf, true));
  v = max(v, (unsigned)__builtin_amdgcn_update_dpp(0, (int)v, 0x112, 0xf, 0xf, true));
  v = max(v, (unsigned)__builtin_amdgcn_update_dpp(0, (int)v, 0x114, 0xf, 0xf, true));
  v = max(v, (unsigned)__builtin_amdgcn_update_dpp(0, (int)v, 0x118, 0xf, 0xf, true));
  v = max(v, (unsigned)__builtin_amdgcn_update_dpp(0, (int)v, 0x142, 0xa, 0xf, true));
  v = max(v, (unsigned)__builtin_amdgcn_update_dpp(0, (int)v, 0x143, 0xc, 0xf, true));
  return (unsigned)__builtin_amdgcn_readlane((int)v, 63);
}

__device__ __forceinline__ int step_k(int t, int rank) {
  if (t < 8) return rank * 512 + t * 64;
  const int u = t - 8, j = (u * 37) >> 8, i = u - j * 7;
  return ((rank + 1 + i) & 7) * 512 + j * 64;
}

__global__ __launch_bounds__(NTHREADS) void a2a_gemm_q8(Params P) {
  extern __shared__ __attribute__((aligned(16))) unsigned char shm[];
  LAS unsigned char* lds = (LAS unsigned char*)shm;
  const unsigned ldsb = (unsigned)(size_t)lds;
  const int tid = threadIdx.x, wid = __builtin_amdgcn_readfirstlane(tid >> 6), lane = tid & 63;
  const int b = blockIdx.x, xcd = b & 7, idx = b >> 3;
  const int ct = xcd * 16 + (idx >> 1), rt = idx & 1;
  const int rank = P.rank;
  char* const myws = P.ws[rank];
  unsigned* const tmo = (unsigned*)(myws + WS_TMO);

  if (wid >= 8) {
    if (wid < 10) {
      const int a = wid - 8;
      unsigned vS[2], vL[4];
#pragma unroll
      for (int h = 0; h < 2; ++h) { const int f = (4 * h + (lane >> 4)) & 7; vS[h] = (unsigned)((lane >> 3) * 128 + (((lane & 7) ^ f) * 16)); }
#pragma unroll
      for (int i = 0; i < 4; ++i) { const int f = (4 * i + (lane >> 4)) & 7; vL[i] = (unsigned)((lane >> 3) * 1024 + (((lane & 7) ^ f) * 16) + i * (8192 - 1024)); }
      const char* xb = P.x + (size_t)(rank * 512 + rt * 256 + a * 128) * 1024;
      const char* sbp = myws + WS_STAGE + (size_t)rt * 32768 + a * 16384;
      const char* flags = myws + WS_FLAGS;
      const unsigned ldst = ldsb + A_OFF + a * 16384;
      const unsigned foff = (unsigned)(lane & 7) * 16;
      auto issue_half = [&](int t, int h) {
        const unsigned l = ldst + (t % NA) * A_ST + h * 8192;
        if (t < 8) { const char* g = xb + t * 128 + h * (64 * 1024);
          dma4(l, g, vL[0], vL[1], vL[2], vL[3]); dma4(l + 4096, g + 32 * 1024, vL[0], vL[1], vL[2], vL[3]); }
        else { const char* g = sbp + (size_t)(t - 8) * 65536 + h * 8192;
          dma4_sys(l, g, vS[0], vS[1], vS[0], vS[1]); dma4_sys(l + 4096, g + 4096, vS[0], vS[1], vS[0], vS[1]); }
      };
      issue_half(0, 0); issue_half(0, 1); issue_half(1, 0); issue_half(1, 1);
      const unsigned seq = sload_u32(myws + WS_SEQ) + 1u;
      WAIT_V(16); BAR;
      int cur_j = -1; unsigned mask = 0; bool dead = false;
      for (int t = 0; t < NT; ++t) {
        const int s = t + 2;
        if (s >= 8 && s < NT && !dead) {
          const int u = s - 8, j = (u * 37) >> 8, src = (rank + 1 + (u - j * 7)) & 7;
          if (j != cur_j || !((mask >> src) & 1u)) {
            cur_j = j;
            const u64 t0 = now();
            for (;;) {
              u32x4 f;
              asm volatile("global_load_dwordx4 %0, %1, %2 sc0 sc1\n\ts_waitcnt vmcnt(0)" : "=&v"(f) : "v"(foff), "s"(flags + j * 128) : "memory");
              const bool ok = f[0] >= seq && f[1] >= seq && f[2] >= seq && f[3] >= seq;
              mask = (unsigned)__builtin_amdgcn_ballot_w64(ok) & 0xffu;
              if ((mask >> src) & 1u) break;
              if (timed_out(tmo, seq) || now() - t0 > DEADLINE_TICKS) { set_timeout(tmo, seq); dead = true; break; }
              __builtin_amdgcn_s_sleep(1);
            }
          }
        }
        bool poll = false; int j1 = 0;
        if (s + 1 >= 8 && s + 1 < NT && !dead) {
          const int u = s + 1 - 8; j1 = (u * 37) >> 8; const int src = (rank + 1 + (u - j1 * 7)) & 7;
          poll = (j1 != cur_j) || !((mask >> src) & 1u);
        }
        const bool live = s < NT && !dead;
        if (poll) {
          u32x4 f;
          asm volatile("global_load_dwordx4 %0, %1, %2 sc0 sc1" : "=&v"(f) : "v"(foff), "s"(flags + j1 * 128) : "memory");
          issue_half(s, 0);
          BAR;
          issue_half(s, 1);
          WAIT_V(16);
          asm volatile("" : "+v"(f));
          const bool ok = f[0] >= seq && f[1] >= seq && f[2] >= seq && f[3] >= seq;
          const unsigned nm = (unsigned)__builtin_amdgcn_ballot_w64(ok) & 0xffu;
          cur_j = j1; mask = nm;
          BAR;
        } else if (live) {
          issue_half(s, 0);
          BAR;
          issue_half(s, 1);
          WAIT_V(16);
          BAR;
        } else {
          BAR;
          WAIT_V(0);
          BAR;
        }
      }
      BAR;
    } else if (wid == 10) {
      unsigned vB[4];
#pragma unroll
      for (int i = 0; i < 4; ++i) { const int q = lane >> 4, s = (q << 1) | ((i >> 1) & 1), u = ((lane & 15) >> 1) ^ s;
        vB[i] = (unsigned)(q * (NN * 2) + (2 * u + (lane & 1)) * 16 + i * (4 * NN * 2 - 1024)); }
      const char* wb = P.w + (size_t)ct * 256;
      const unsigned ldst = ldsb + B_OFF;
      auto issue_half = [&](int t, int h) {
        const char* g = wb + (size_t)(step_k(t, rank) + h * 32) * (NN * 2);
        const unsigned l = ldst + (t % NB) * B_ST + h * 8192;
        dma4(l, g, vB[0], vB[1], vB[2], vB[3]); dma4(l + 4096, g + (size_t)16 * NN * 2, vB[0], vB[1], vB[2], vB[3]);
      };
      issue_half(0, 0); issue_half(0, 1); issue_half(1, 0); issue_half(1, 1); issue_half(2, 0); issue_half(2, 1);
      WAIT_V(32); BAR;
      for (int t = 0; t < NT - 3; ++t) {
        issue_half(t + 3, 0);
        BAR;
        issue_half(t + 3, 1);
        WAIT_V(32);
        BAR;
      }
      BAR; WAIT_V(16); BAR;
      BAR; WAIT_V(0); BAR;
      BAR; BAR;
      BAR;
    } else {
      if (b >= 224) return;
      const int j = b / 28, r = b - j * 28, i = r >> 2, q = r & 3;
      const int dst = (rank + 1 + i) & 7, u = 7 * j + (6 - i);
      const char* src = P.x + (size_t)(dst * 512 + q * 128) * 1024 + j * 128;
      char* dws = P.ws[dst];
      char* dbase = dws + WS_STAGE + (size_t)u * 65536 + q * 16384;
      const unsigned voff = (unsigned)((lane >> 3) * 1024 + (lane & 7) * 16);
      u32x4 d[16];
#pragma unroll
      for (int k = 0; k < 16; ++k)
        asm volatile("global_load_dwordx4 %0, %1, %2" : "=&v"(d[k]) : "v"(voff), "s"(src + k * 8192) : "memory");
      const unsigned seq = sload_u32(myws + WS_SEQ) + 1u;
      const int start = PACE * j;
      for (int s = 0; s < start; ++s) BAR;
      WAIT_V(0);
      const unsigned soff = (unsigned)lane * 16;
#pragma unroll
      for (int k = 0; k < 16; ++k)
        asm volatile("global_store_dwordx4 %0, %1, %2 sc0 sc1\n\ts_nop 1" :: "v"(soff), "v"(d[k]), "s"(dbase + k * 1024) : "memory");
      for (int s = 0; s < PUB_DELAY; ++s) BAR;
      WAIT_V(0);
      if (lane == 0)
        __hip_atomic_store((unsigned*)(dws + WS_FLAGS + j * 128 + rank * 16 + q * 4), seq, __ATOMIC_RELAXED, __HIP_MEMORY_SCOPE_SYSTEM);
    }
    return;
  }

  const int grp = wid >> 2, wm = wid & 3, wn = grp;
  const int fr = lane & 15, fq = lane >> 4;
  unsigned aoff[2];
#pragma unroll
  for (int kh = 0; kh < 2; ++kh) aoff[kh] = ldsb + (unsigned)(A_OFF + (wm * 64 + fr) * 128 + (((4 * kh + fq) ^ ((fr >> 1) & 7)) * 16));
  unsigned boff[4];
  { const int q = (lane & 15) >> 2, p = lane & 3; const int s = (q << 1) | (fq & 1);
#pragma unroll
    for (int n = 0; n < 4; ++n) { const int u = (wn * 4 + n) ^ s; boff[n] = ldsb + (unsigned)(B_OFF + (8 * fq + q) * 256 + u * 32 + p * 8); } }
  unsigned aoffH[2] = {aoff[0] + 65536u, aoff[1] + 65536u};
  asm volatile("" : "+v"(aoffH[0]), "+v"(aoffH[1]));

  f32x4 acc[4][4];
#pragma unroll
  for (int m = 0; m < 4; ++m)
#pragma unroll
    for (int n = 0; n < 4; ++n) acc[m][n] = (f32x4){0.f, 0.f, 0.f, 0.f};
  bf16x8 af[4][2], bfr[4][2];
  const unsigned seq = sload_u32(myws + WS_SEQ) + 1u;

  BAR;
  if (grp == 1) BAR;

#define TRD(dst, addr, off) asm volatile("ds_read_b64_tr_b16 %0, %1 offset:%2" : "=&v"(dst) : "v"(addr), "i"(off) : "memory")
#define ARD(dst, addr, off) asm volatile("ds_read_b128 %0, %1 offset:%2" : "=&v"(dst) : "v"(addr), "i"(off) : "memory")
  auto step = [&](auto SAc, auto SBc) {
    constexpr int SA = decltype(SAc)::value, SB = decltype(SBc)::value;
    bf16x4 lo[4][2], hi[4][2];
#pragma unroll
    for (int n = 0; n < 4; ++n) {
      TRD(lo[n][0], boff[n], SB * B_ST); TRD(hi[n][0], boff[n], SB * B_ST + 1024);
      TRD(lo[n][1], boff[n], SB * B_ST + 8192); TRD(hi[n][1], boff[n], SB * B_ST + 9216);
    }
#pragma unroll
    for (int m = 0; m < 4; ++m) {
      if constexpr (SA < 2) { ARD(af[m][0], aoff[0], SA * A_ST + m * 2048); ARD(af[m][1], aoff[1], SA * A_ST + m * 2048); }
      else { ARD(af[m][0], aoffH[0], m * 2048); ARD(af[m][1], aoffH[1], m * 2048); }
    }
    WAIT_L(0);
    SCHED;
#pragma unroll
    for (int n = 0; n < 4; ++n)
#pragma unroll
      for (int kh = 0; kh < 2; ++kh) bfr[n][kh] = __builtin_shufflevector(lo[n][kh], hi[n][kh], 0, 1, 2, 3, 4, 5, 6, 7);
    BAR;
    __builtin_amdgcn_s_setprio(1);
#pragma unroll
    for (int kh = 0; kh < 2; ++kh)
#pragma unroll
      for (int m = 0; m < 4; ++m)
#pragma unroll
        for (int n = 0; n < 4; ++n) acc[m][n] = __builtin_amdgcn_mfma_f32_16x16x32_bf16(bfr[n][kh], af[m][kh], acc[m][n], 0, 0, 0);
    __builtin_amdgcn_s_setprio(0);
    BAR;
    SCHED;
  };
  for (int it = 0; it < 5; ++it) {
    step(IC(0), IC(0)); step(IC(1), IC(1)); step(IC(2), IC(2)); step(IC(0), IC(3));
    step(IC(1), IC(0)); step(IC(2), IC(1)); step(IC(0), IC(2)); step(IC(1), IC(3));
    step(IC(2), IC(0)); step(IC(0), IC(1)); step(IC(1), IC(2)); step(IC(2), IC(3));
  }
  step(IC(0), IC(0)); step(IC(1), IC(1)); step(IC(2), IC(2)); step(IC(0), IC(3));

  float mx = 0.f;
#pragma unroll
  for (int m = 0; m < 4; ++m)
#pragma unroll
    for (int n = 0; n < 4; ++n) {
      mx = __builtin_fmaxf(mx, __builtin_fmaxf(__builtin_fabsf(acc[m][n][0]), __builtin_fabsf(acc[m][n][1])));
      mx = __builtin_fmaxf(mx, __builtin_fmaxf(__builtin_fabsf(acc[m][n][2]), __builtin_fabsf(acc[m][n][3])));
    }
  const unsigned wmx = wave_max_u32(__float_as_uint(mx));
  if (grp == 0) BAR;
  LAS unsigned* red = (LAS unsigned*)lds;
  if (lane == 0) red[wid] = wmx;
  WAIT_L(0); BAR;

  if (wid == 0) {
    unsigned v = red[lane & 7];
    v = max(v, (unsigned)__builtin_amdgcn_update_dpp(0, (int)v, 0x111, 0xf, 0xf, true));
    v = max(v, (unsigned)__builtin_amdgcn_update_dpp(0, (int)v, 0x112, 0xf, 0xf, true));
    v = max(v, (unsigned)__builtin_amdgcn_update_dpp(0, (int)v, 0x114, 0xf, 0xf, true));
    const unsigned gmx = (unsigned)__builtin_amdgcn_readlane((int)v, 7);
    u64* lg = (u64*)(myws + WS_LGRAN);
    if (lane == 0) __hip_atomic_store(lg + b, ((u64)seq << 32) | gmx, __ATOMIC_RELAXED, __HIP_MEMORY_SCOPE_AGENT);
    bool dead = false;
    if (b < NRANK) {
      const u64 t0 = now(); unsigned am = 0;
      for (;;) {
        bool ok = true; am = 0;
#pragma unroll
        for (int k = 0; k < 4; ++k) { const u64 g = __hip_atomic_load(lg + k * 64 + lane, __ATOMIC_RELAXED, __HIP_MEMORY_SCOPE_AGENT);
          ok = ok && (unsigned)(g >> 32) >= seq; am = max(am, (unsigned)g); }
        if (__builtin_amdgcn_ballot_w64(ok) == ~0ull) break;
        if (timed_out(tmo, seq) || now() - t0 > DEADLINE_TICKS) { set_timeout(tmo, seq); dead = true; break; }
        __builtin_amdgcn_s_sleep(1);
      }
      am = wave_max_u32(am);
      if (!dead && lane < NREP)
        __hip_atomic_store((u64*)(P.ws[b] + WS_GGRAN + lane * 128 + rank * 8), ((u64)seq << 32) | am, __ATOMIC_RELAXED, __HIP_MEMORY_SCOPE_SYSTEM);
    }
    const u64* gg = (const u64*)(myws + WS_GGRAN + (b & (NREP - 1)) * 128) + (lane & 7);
    const u64 t0 = now(); unsigned am = 0;
    while (!dead) {
      const u64 g = __hip_atomic_load(gg, __ATOMIC_RELAXED, __HIP_MEMORY_SCOPE_SYSTEM);
      am = (unsigned)g;
      if (__builtin_amdgcn_ballot_w64((unsigned)(g >> 32) >= seq) == ~0ull) break;
      if (timed_out(tmo, seq) || now() - t0 > DEADLINE_TICKS) { set_timeout(tmo, seq); dead = true; break; }
      __builtin_amdgcn_s_sleep(1);
    }
    am = max(am, (unsigned)__builtin_amdgcn_update_dpp(0, (int)am, 0x111, 0xf, 0xf, true));
    am = max(am, (unsigned)__builtin_amdgcn_update_dpp(0, (int)am, 0x112, 0xf, 0xf, true));
    am = max(am, (unsigned)__builtin_amdgcn_update_dpp(0, (int)am, 0x114, 0xf, 0xf, true));
    am = (unsigned)__builtin_amdgcn_readlane((int)am, 7);
    const bool bad = dead || timed_out(tmo, seq);
    if (lane == 0) { red[8] = am; red[9] = bad ? 1u : 0u;
      if (b == 0) __hip_atomic_store((unsigned*)(myws + WS_SEQ), seq, __ATOMIC_RELAXED, __HIP_MEMORY_SCOPE_AGENT); }
  }
  WAIT_L(0); BAR;
  const float amax = __uint_as_float(red[8]);
  const bool bad = red[9] != 0u;
  const float inv = amax > 0.f ? 1.0f / (amax / 127.0f) : 0.f;

  signed char* orow = P.out + (size_t)(rt * 256 + wm * 64 + fr) * NN + ct * 128 + wn * 64 + fq * 16;
#pragma unroll
  for (int m = 0; m < 4; ++m) {
    unsigned d[4];
#pragma unroll
    for (int n = 0; n < 4; ++n) {
      unsigned q[4];
#pragma unroll
      for (int e = 0; e < 4; ++e) q[e] = __float_as_uint(__builtin_fmaf(acc[m][n][e], inv, 12582912.0f));
      const unsigned p01 = __builtin_amdgcn_perm(q[1], q[0], 0x0c0c0400u), p23 = __builtin_amdgcn_perm(q[3], q[2], 0x04000c0cu);
      d[n] = p01 | p23;
    }
    auto r0 = __builtin_amdgcn_permlane32_swap(d[0], d[2], false, false); d[0] = r0[0]; d[2] = r0[1];
    auto r1 = __builtin_amdgcn_permlane32_swap(d[1], d[3], false, false); d[1] = r1[0]; d[3] = r1[1];
    auto r2 = __builtin_amdgcn_permlane16_swap(d[0], d[1], false, false); d[0] = r2[0]; d[1] = r2[1];
    auto r3 = __builtin_amdgcn_permlane16_swap(d[2], d[3], false, false); d[2] = r3[0]; d[3] = r3[1];
    u32x4 o = (u32x4){d[0], d[1], d[2], d[3]};
    if (bad) o = (u32x4){0x80808080u, 0x80808080u, 0x80808080u, 0x80808080u};
    *(u32x4*)(orow + (size_t)m * 16 * NN) = o;
  }
}

}

extern "C" void kernel_launch(int rank, int nranks, ncclComm_t comm,
                              hipStream_t stream,
                              void* const* d_in, const int* in_sizes, int n_in,
                              void* d_out, int out_size,
                              const int* in_dtypes, int out_dtype,
                              void* const* d_out_all,
                              void* const* d_ws_all, size_t ws_bytes) {
  (void)comm; (void)in_dtypes; (void)out_dtype; (void)d_out_all;
  if (nranks != NRANK || (unsigned)rank >= (unsigned)NRANK || n_in != 2 || ws_bytes < WS_NEED ||
      in_sizes[0] != 4096 * 512 || in_sizes[1] != KK * NN || out_size != 512 * NN) {
    fprintf(stderr, "kernel_launch: unexpected problem shape\n");
    return;
  }
  static bool init = false;
  if (!init) {
    HIP_CHECK(hipFuncSetAttribute((const void*)a2a_gemm_q8, hipFuncAttributeMaxDynamicSharedMemorySize, LDS_BYTES));
    init = true;
  }
  Params P;
  P.x = (const char*)d_in[0];
  P.w = (const char*)d_in[1];
  P.out = (signed char*)d_out;
  for (int p = 0; p < NRANK; ++p) P.ws[p] = (char*)d_ws_all[p];
  P.rank = rank;
  hipLaunchKernelGGL(a2a_gemm_q8, dim3(NWG), dim3(NTHREADS), LDS_BYTES, stream, P);
  HIP_CHECK(hipGetLastError());
}
```

```cpp
#include <hip/hip_runtime.h>
#include <rccl/rccl.h>
#include <cstdio>
#include <type_traits>

#define HIP_CHECK(expr) do { hipError_t _e = (expr); if (_e != hipSuccess) { \
    fprintf(stderr, "HIP error %s at %s:%d\n", hipGetErrorString(_e), __FILE__, __LINE__); \
    return; } } while (0)

#define LAS __attribute__((address_space(3)))
#define CAS __attribute__((address_space(4)))
typedef short bf16x8 __attribute__((ext_vector_type(8)));
typedef short bf16x4 __attribute__((ext_vector_type(4)));
typedef float f32x4 __attribute__((ext_vector_type(4)));
typedef unsigned u32x4 __attribute__((ext_vector_type(4)));
typedef unsigned long long u64;

namespace {

constexpr int NRANK = 8;
constexpr int NN = 16384, KK = 4096;
constexpr int NT = KK / 64;
constexpr int A_ST = 32768, B_ST = 16384, NA = 3, NB = 4;
constexpr int B_OFF = 0, A_OFF = NB * B_ST, LDS_BYTES = NA * A_ST + NB * B_ST;
constexpr int NTHREADS = 768, NWG = 256;

constexpr size_t WS_SEQ = 0, WS_TMO = 128;
constexpr size_t WS_FLAGS = 4096;
constexpr size_t WS_LGRAN = 8192;
constexpr size_t WS_GGRAN = 16384;
constexpr int NREP = 32;
constexpr size_t WS_STAGE = 1u << 20;
constexpr size_t WS_NEED = WS_STAGE + (size_t)56 * 65536;

constexpr u64 DEADLINE_TICKS = 100000000ull;
#define PACE 10

struct Params {
  const char* x;
  const char* w;
  signed char* out;
  char* ws[NRANK];
  int rank;
};
#define STAMPC(i) do { } while (0)
#define STAMP(i) do { } while (0)

#define WAIT_V(n) asm volatile("s_waitcnt vmcnt(" #n ")" ::: "memory")
#define WAIT_L(n) asm volatile("s_waitcnt lgkmcnt(" #n ")" ::: "memory")
#define BAR __builtin_amdgcn_s_barrier()
#define SCHED __builtin_amdgcn_sched_barrier(0)
#define IC(x) std::integral_constant<int, (x)>{}

#define DEF_DMA4(NAME, POL) \
__device__ __forceinline__ void NAME(unsigned lds_addr, const char* gbase, unsigned v0, unsigned v1, unsigned v2, unsigned v3) { \
  asm volatile("s_mov_b32 m0, %0\n\ts_nop 0\n\t" \
               "global_load_lds_dwordx4 %2, %1" POL "\n\t" \
               "global_load_lds_dwordx4 %3, %1 offset:1024" POL "\n\t" \
               "global_load_lds_dwordx4 %4, %1 offset:2048" POL "\n\t" \
               "global_load_lds_dwordx4 %5, %1 offset:3072" POL \
               :: "s"(lds_addr), "s"(gbase), "v"(v0), "v"(v1), "v"(v2), "v"(v3) : "memory"); \
}
#define WPOL ""
DEF_DMA4(dma4, "")
DEF_DMA4(dma4_w, WPOL)
DEF_DMA4(dma4_sys, " sc0 sc1")

__device__ __forceinline__ unsigned sload_u32(const char* p) { return *(const CAS unsigned*)p; }
__device__ __forceinline__ bool vm_busy() { unsigned r; asm volatile("s_getreg_b32 %0, hwreg(HW_REG_IB_STS)" : "=s"(r) :: "memory"); return (r & 0x00c0000fu) != 0u; }
__device__ __forceinline__ u64 now() { return __builtin_amdgcn_s_memrealtime(); }
__device__ __forceinline__ bool timed_out(unsigned* tmo, unsigned seq) {
  return (unsigned)__builtin_amdgcn_readfirstlane(__hip_atomic_load(tmo, __ATOMIC_RELAXED, __HIP_MEMORY_SCOPE_AGENT)) == seq;
}
__device__ __forceinline__ void set_timeout(unsigned* tmo, unsigned seq) { __hip_atomic_store(tmo, seq, __ATOMIC_RELAXED, __HIP_MEMORY_SCOPE_AGENT); }

__device__ __forceinline__ unsigned wave_max_u32(unsigned v) {
  v = max(v, (unsigned)__builtin_amdgcn_update_dpp(0, (int)v, 0x111, 0xf, 0xf, true));
  v = max(v, (unsigned)__builtin_amdgcn_update_dpp(0, (int)v, 0x112, 0xf, 0xf, true));
  v = max(v, (unsigned)__builtin_amdgcn_update_dpp(0, (int)v, 0x114, 0xf, 0xf, true));
  v = max(v, (unsigned)__builtin_amdgcn_update_dpp(0, (int)v, 0x118, 0xf, 0xf, true));
  v = max(v, (unsigned)__builtin_amdgcn_update_dpp(0, (int)v, 0x142, 0xa, 0xf, true));
  v = max(v, (unsigned)__builtin_amdgcn_update_dpp(0, (int)v, 0x143, 0xc, 0xf, true));
  return (unsigned)__builtin_amdgcn_readlane((int)v, 63);
}

__device__ __forceinline__ int step_k(int t, int rank) {
  if (t < 8) return rank * 512 + t * 64;
  const int u = t - 8, j = (u * 37) >> 8, i = u - j * 7;
  return ((rank + 1 + i) & 7) * 512 + j * 64;
}

__global__ __launch_bounds__(NTHREADS) void a2a_gemm_q8(Params P) {
  extern __shared__ __attribute__((aligned(16))) unsigned char shm[];
  LAS unsigned char* lds = (LAS unsigned char*)shm;
  const unsigned ldsb = (unsigned)(size_t)lds;
  const int tid = threadIdx.x, wid = __builtin_amdgcn_readfirstlane(tid >> 6), lane = tid & 63;
  const int b = blockIdx.x, xcd = b & 7, idx = b >> 3;
  const int ct = xcd * 16 + (idx >> 1), rt = idx & 1;
  const int rank = P.rank;
  char* const myws = P.ws[rank];
  unsigned* const tmo = (unsigned*)(myws + WS_TMO);

  if (wid >= 8) {
    if (wid < 10) {
      const int a = wid - 8;
      unsigned vS[2], vL[4];
#pragma unroll
      for (int h = 0; h < 2; ++h) { const int f = (4 * h + (lane >> 4)) & 7; vS[h] = (unsigned)((lane >> 3) * 128 + (((lane & 7) ^ f) * 16)); }
#pragma unroll
      for (int i = 0; i < 4; ++i) { const int f = (4 * i + (lane >> 4)) & 7; vL[i] = (unsigned)((lane >> 3) * 1024 + (((lane & 7) ^ f) * 16) + i * (8192 - 1024)); }
      const char* xb = P.x + (size_t)(rank * 512 + rt * 256 + a * 128) * 1024;
      const char* sbp = myws + WS_STAGE + (size_t)rt * 32768 + a * 16384;
      const char* flags = myws + WS_FLAGS;
      const unsigned ldst = ldsb + A_OFF + a * 16384;
      const unsigned foff = (unsigned)(lane & 7) * 16;
      auto issue_half = [&](int t, int h) {
        const unsigned l = ldst + (t % NA) * A_ST + h * 8192;
        if (t < 8) { const char* g = xb + t * 128 + h * (64 * 1024);
          dma4(l, g, vL[0], vL[1], vL[2], vL[3]); dma4(l + 4096, g + 32 * 1024, vL[0], vL[1], vL[2], vL[3]); }
        else { const char* g = sbp + (size_t)(t - 8) * 65536 + h * 8192;
          dma4_sys(l, g, vS[0], vS[1], vS[0], vS[1]); dma4_sys(l + 4096, g + 4096, vS[0], vS[1], vS[0], vS[1]); }
      };
      issue_half(0, 0); issue_half(0, 1); issue_half(1, 0); issue_half(1, 1);
      const unsigned seq = sload_u32(myws + WS_SEQ) + 1u;
      WAIT_V(16); BAR;
      int cur_j = -1; unsigned mask = 0; bool dead = false;
      for (int t = 0; t < NT; ++t) {
        const int s = t + 2;
        if (s >= 8 && s < NT && !dead) {
          const int u = s - 8, j = (u * 37) >> 8, src = (rank + 1 + (u - j * 7)) & 7;
          if (j != cur_j || !((mask >> src) & 1u)) {
            cur_j = j;
            const u64 t0 = now();
            if (a == 0) STAMP(64 + (s - 8));
            for (;;) {
              u32x4 f;
              asm volatile("global_load_dwordx4 %0, %1, %2 sc0 sc1\n\ts_waitcnt vmcnt(0)" : "=&v"(f) : "v"(foff), "s"(flags + j * 128) : "memory");
              const bool ok = f[0] >= seq && f[1] >= seq && f[2] >= seq && f[3] >= seq;
              mask = (unsigned)__builtin_amdgcn_ballot_w64(ok) & 0xffu;
              if ((mask >> src) & 1u) break;
              if (timed_out(tmo, seq) || now() - t0 > DEADLINE_TICKS) { set_timeout(tmo, seq); dead = true; break; }
              __builtin_amdgcn_s_sleep(1);
            }
          }
        }
        bool poll = false; int j1 = 0;
        if (s + 1 >= 8 && s + 1 < NT && !dead) {
          const int u = s + 1 - 8; j1 = (u * 37) >> 8; const int src = (rank + 1 + (u - j1 * 7)) & 7;
          poll = (j1 != cur_j) || !((mask >> src) & 1u);
        }
        const bool live = s < NT && !dead;
        if (poll) {
          u32x4 f;
          asm volatile("global_load_dwordx4 %0, %1, %2 sc0 sc1" : "=&v"(f) : "v"(foff), "s"(flags + j1 * 128) : "memory");
          issue_half(s, 0);
          BAR;
          issue_half(s, 1);
          WAIT_V(16);
          asm volatile("" : "+v"(f));
          const bool ok = f[0] >= seq && f[1] >= seq && f[2] >= seq && f[3] >= seq;
          const unsigned nm = (unsigned)__builtin_amdgcn_ballot_w64(ok) & 0xffu;
          cur_j = j1; mask = nm;
          BAR;
        } else if (live) {
          issue_half(s, 0);
          BAR;
          issue_half(s, 1);
          WAIT_V(16);
          BAR;
        } else {
          BAR;
          WAIT_V(0);
          BAR;
        }
      }
      BAR;
    } else if (wid == 10) {
      unsigned vB[4];
#pragma unroll
      for (int i = 0; i < 4; ++i) { const int q = lane >> 4, s = (q << 1) | ((i >> 1) & 1), u = ((lane & 15) >> 1) ^ s;
        vB[i] = (unsigned)(q * (NN * 2) + (2 * u + (lane & 1)) * 16 + i * (4 * NN * 2 - 1024)); }
      const char* wb = P.w + (size_t)ct * 256;
      const unsigned ldst = ldsb + B_OFF;
      auto issue_half = [&](int t, int h) {
        const char* g = wb + (size_t)(step_k(t, rank) + h * 32) * (NN * 2);
        const unsigned l = ldst + (t % NB) * B_ST + h * 8192;
        dma4_w(l, g, vB[0], vB[1], vB[2], vB[3]); dma4_w(l + 4096, g + (size_t)16 * NN * 2, vB[0], vB[1], vB[2], vB[3]);
      };
      issue_half(0, 0); issue_half(0, 1); issue_half(1, 0); issue_half(1, 1); issue_half(2, 0); issue_half(2, 1);
      WAIT_V(32); BAR;
      for (int t = 0; t < NT - 3; ++t) {
        issue_half(t + 3, 0);
        BAR;
        issue_half(t + 3, 1);
        WAIT_V(32);
        BAR;
      }
      BAR; WAIT_V(16); BAR;
      BAR; WAIT_V(0); BAR;
      BAR; BAR;
      BAR;
    } else {
      if (b >= 224) return;
      const int j = b / 28, r = b - j * 28, i = r >> 2, q = r & 3;
      const int dst = (rank + 1 + i) & 7, u = 7 * j + (6 - i);
      const char* src = P.x + (size_t)(dst * 512 + q * 128) * 1024 + j * 128;
      char* dws = P.ws[dst];
      char* dbase = dws + WS_STAGE + (size_t)u * 65536 + q * 16384;
      const unsigned voff = (unsigned)((lane >> 3) * 1024 + (lane & 7) * 16);
      u32x4 d[16];
#pragma unroll
      for (int k = 0; k < 16; ++k)
        asm volatile("global_load_dwordx4 %0, %1, %2" : "=&v"(d[k]) : "v"(voff), "s"(src + k * 8192) : "memory");
      const unsigned seq = sload_u32(myws + WS_SEQ) + 1u;
      const int start = PACE * j;
      int s = 0;
      while (s < start || vm_busy()) { BAR; ++s; }
      WAIT_V(0);
      STAMP(32);
      const unsigned soff = (unsigned)lane * 16;
#pragma unroll
      for (int k = 0; k < 16; ++k)
        asm volatile("global_store_dwordx4 %0, %1, %2 sc0 sc1\n\ts_nop 1" :: "v"(soff), "v"(d[k]), "s"(dbase + k * 1024) : "memory");
      while (vm_busy()) BAR;
      WAIT_V(0);
      STAMP(34);
      if (lane == 0)
        __hip_atomic_store((unsigned*)(dws + WS_FLAGS + j * 128 + rank * 16 + q * 4), seq, __ATOMIC_RELAXED, __HIP_MEMORY_SCOPE_SYSTEM);
    }
    return;
  }

  const int grp = wid >> 2, wm = wid & 3, wn = grp;
  const int fr = lane & 15, fq = lane >> 4;
  unsigned aoff[2];
#pragma unroll
  for (int kh = 0; kh < 2; ++kh) aoff[kh] = ldsb + (unsigned)(A_OFF + (wm * 64 + fr) * 128 + (((4 * kh + fq) ^ ((fr >> 1) & 7)) * 16));
  unsigned boff[4];
  { const int q = (lane & 15) >> 2, p = lane & 3; const int s = (q << 1) | (fq & 1);
#pragma unroll
    for (int n = 0; n < 4; ++n) { const int u = (wn * 4 + n) ^ s; boff[n] = ldsb + (unsigned)(B_OFF + (8 * fq + q) * 256 + u * 32 + p * 8); } }
  unsigned aoffH[2] = {aoff[0] + 65536u, aoff[1] + 65536u};
  asm volatile("" : "+v"(aoffH[0]), "+v"(aoffH[1]));

  f32x4 acc[4][4];
#pragma unroll
  for (int m = 0; m < 4; ++m)
#pragma unroll
    for (int n = 0; n < 4; ++n) acc[m][n] = (f32x4){0.f, 0.f, 0.f, 0.f};
  bf16x8 af[4][2], bfr[4][2];
  const unsigned seq = sload_u32(myws + WS_SEQ) + 1u;

  if (wid == 0) STAMP(0);
  BAR;
  if (grp == 1) BAR;
  if (wid == 0) STAMP(1);

#define TRD(dst, addr, off) asm volatile("ds_read_b64_tr_b16 %0, %1 offset:%2" : "=&v"(dst) : "v"(addr), "i"(off) : "memory")
#define ARD(dst, addr, off) asm volatile("ds_read_b128 %0, %1 offset:%2" : "=&v"(dst) : "v"(addr), "i"(off) : "memory")
  auto step = [&](auto SAc, auto SBc) {
    constexpr int SA = decltype(SAc)::value, SB = decltype(SBc)::value;
    bf16x4 lo[4][2], hi[4][2];
#pragma unroll
    for (int n = 0; n < 4; ++n) {
      TRD(lo[n][0], boff[n], SB * B_ST); TRD(hi[n][0], boff[n], SB * B_ST + 1024);
      TRD(lo[n][1], boff[n], SB * B_ST + 8192); TRD(hi[n][1], boff[n], SB * B_ST + 9216);
    }
#pragma unroll
    for (int m = 0; m < 4; ++m) {
      if constexpr (SA < 2) { ARD(af[m][0], aoff[0], SA * A_ST + m * 2048); ARD(af[m][1], aoff[1], SA * A_ST + m * 2048); }
      else { ARD(af[m][0], aoffH[0], m * 2048); ARD(af[m][1], aoffH[1], m * 2048); }
    }
    WAIT_L(0);
    SCHED;
#pragma unroll
    for (int n = 0; n < 4; ++n)
#pragma unroll
      for (int kh = 0; kh < 2; ++kh) bfr[n][kh] = __builtin_shufflevector(lo[n][kh], hi[n][kh], 0, 1, 2, 3, 4, 5, 6, 7);
    BAR;
    __builtin_amdgcn_s_setprio(1);
#pragma unroll
    for (int kh = 0; kh < 2; ++kh)
#pragma unroll
      for (int m = 0; m < 4; ++m)
#pragma unroll
        for (int n = 0; n < 4; ++n) acc[m][n] = __builtin_amdgcn_mfma_f32_16x16x32_bf16(bfr[n][kh], af[m][kh], acc[m][n], 0, 0, 0);
    __builtin_amdgcn_s_setprio(0);
    BAR;
    SCHED;
  };
  for (int it = 0; it < 5; ++it) {
    step(IC(0), IC(0)); step(IC(1), IC(1)); step(IC(2), IC(2)); step(IC(0), IC(3));
    if (wid == 0) { STAMP(8 + 3 * it); STAMPC(40 + it); }
    step(IC(1), IC(0)); step(IC(2), IC(1)); step(IC(0), IC(2)); step(IC(1), IC(3));
    if (wid == 0) STAMP(9 + 3 * it);
    step(IC(2), IC(0)); step(IC(0), IC(1)); step(IC(1), IC(2)); step(IC(2), IC(3));
    if (wid == 0) STAMP(10 + 3 * it);
  }
  step(IC(0), IC(0)); step(IC(1), IC(1)); step(IC(2), IC(2)); step(IC(0), IC(3));

  if (wid == 0) STAMP(2);
  float mx = 0.f;
#pragma unroll
  for (int m = 0; m < 4; ++m)
#pragma unroll
    for (int n = 0; n < 4; ++n) {
      mx = __builtin_fmaxf(mx, __builtin_fmaxf(__builtin_fabsf(acc[m][n][0]), __builtin_fabsf(acc[m][n][1])));
      mx = __builtin_fmaxf(mx, __builtin_fmaxf(__builtin_fabsf(acc[m][n][2]), __builtin_fabsf(acc[m][n][3])));
    }
  const unsigned wmx = wave_max_u32(__float_as_uint(mx));
  if (grp == 0) BAR;
  LAS unsigned* red = (LAS unsigned*)lds;
  if (lane == 0) red[wid] = wmx;
  WAIT_L(0); BAR;

  if (wid == 0) {
    unsigned v = red[lane & 7];
    v = max(v, (unsigned)__builtin_amdgcn_update_dpp(0, (int)v, 0x111, 0xf, 0xf, true));
    v = max(v, (unsigned)__builtin_amdgcn_update_dpp(0, (int)v, 0x112, 0xf, 0xf, true));
    v = max(v, (unsigned)__builtin_amdgcn_update_dpp(0, (int)v, 0x114, 0xf, 0xf, true));
    const unsigned gmx = (unsigned)__builtin_amdgcn_readlane((int)v, 7);
    u64* lg = (u64*)(myws + WS_LGRAN);
    if (lane == 0) __hip_atomic_store(lg + b, ((u64)seq << 32) | gmx, __ATOMIC_RELAXED, __HIP_MEMORY_SCOPE_AGENT);
    STAMP(3);
    bool dead = false;
    if (b < NRANK) {
      const u64 t0 = now(); unsigned am = 0;
      for (unsigned it = 1;; ++it) {
        bool ok = true; am = 0;
#pragma unroll
        for (int k = 0; k < 4; ++k) { const u64 g = __hip_atomic_load(lg + k * 64 + lane, __ATOMIC_RELAXED, __HIP_MEMORY_SCOPE_AGENT);
          ok = ok && (unsigned)(g >> 32) >= seq; am = max(am, (unsigned)g); }
        if (__builtin_amdgcn_ballot_w64(ok) == ~0ull) break;
        if ((it & 15u) == 0u && (timed_out(tmo, seq) || now() - t0 > DEADLINE_TICKS)) { set_timeout(tmo, seq); dead = true; break; }
      }
      am = wave_max_u32(am);
      if (!dead && lane < NREP)
        __hip_atomic_store((u64*)(P.ws[b] + WS_GGRAN + lane * 128 + rank * 8), ((u64)seq << 32) | am, __ATOMIC_RELAXED, __HIP_MEMORY_SCOPE_SYSTEM);
    }
    STAMP(4);
    const u64* gg = (const u64*)(myws + WS_GGRAN + (b & (NREP - 1)) * 128) + (lane & 7);
    const u64 t0 = now(); unsigned am = 0;
    for (unsigned it = 1; !dead; ++it) {
      const u64 g = __hip_atomic_load(gg, __ATOMIC_RELAXED, __HIP_MEMORY_SCOPE_SYSTEM);
      am = (unsigned)g;
      if (__builtin_amdgcn_ballot_w64((unsigned)(g >> 32) >= seq) == ~0ull) break;
      if ((it & 15u) == 0u && (timed_out(tmo, seq) || now() - t0 > DEADLINE_TICKS)) { set_timeout(tmo, seq); dead = true; break; }
      __builtin_amdgcn_s_sleep(1);
    }
    am = max(am, (unsigned)__builtin_amdgcn_update_dpp(0, (int)am, 0x111, 0xf, 0xf, true));
    am = max(am, (unsigned)__builtin_amdgcn_update_dpp(0, (int)am, 0x112, 0xf, 0xf, true));
    am = max(am, (unsigned)__builtin_amdgcn_update_dpp(0, (int)am, 0x114, 0xf, 0xf, true));
    am = (unsigned)__builtin_amdgcn_readlane((int)am, 7);
    STAMP(5);
    const bool bad = dead || timed_out(tmo, seq);
    if (lane == 0) { red[8] = am; red[9] = bad ? 1u : 0u;
      if (b == 0) __hip_atomic_store((unsigned*)(myws + WS_SEQ), seq, __ATOMIC_RELAXED, __HIP_MEMORY_SCOPE_AGENT); }
  }
  WAIT_L(0); BAR;
  const float amax = __uint_as_float(red[8]);
  const bool bad = red[9] != 0u;
  const float inv = amax > 0.f ? 1.0f / (amax / 127.0f) : 0.f;

  signed char* orow = P.out + (size_t)(rt * 256 + wm * 64 + fr) * NN + ct * 128 + wn * 64 + fq * 16;
#pragma unroll
  for (int m = 0; m < 4; ++m) {
    unsigned d[4];
#pragma unroll
    for (int n = 0; n < 4; ++n) {
      unsigned q[4];
#pragma unroll
      for (int e = 0; e < 4; ++e) q[e] = __float_as_uint(__builtin_fmaf(acc[m][n][e], inv, 12582912.0f));
      const unsigned p01 = __builtin_amdgcn_perm(q[1], q[0], 0x0c0c0400u), p23 = __builtin_amdgcn_perm(q[3], q[2], 0x04000c0cu);
      d[n] = p01 | p23;
    }
    auto r0 = __builtin_amdgcn_permlane32_swap(d[0], d[2], false, false); d[0] = r0[0]; d[2] = r0[1];
    auto r1 = __builtin_amdgcn_permlane32_swap(d[1], d[3], false, false); d[1] = r1[0]; d[3] = r1[1];
    auto r2 = __builtin_amdgcn_permlane16_swap(d[0], d[1], false, false); d[0] = r2[0]; d[1] = r2[1];
    auto r3 = __builtin_amdgcn_permlane16_swap(d[2], d[3], false, false); d[2] = r3[0]; d[3] = r3[1];
    u32x4 o = (u32x4){d[0], d[1], d[2], d[3]};
    if (bad) o = (u32x4){0x80808080u, 0x80808080u, 0x80808080u, 0x80808080u};
    *(u32x4*)(orow + (size_t)m * 16 * NN) = o;
  }
  if (wid == 0) STAMP(6);
}

}

extern "C" void kernel_launch(int rank, int nranks, ncclComm_t comm,
                              hipStream_t stream,
                              void* const* d_in, const int* in_sizes, int n_in,
                              void* d_out, int out_size,
                              const int* in_dtypes, int out_dtype,
                              void* const* d_out_all,
                              void* const* d_ws_all, size_t ws_bytes) {
  (void)comm; (void)in_dtypes; (void)out_dtype; (void)d_out_all;
  if (nranks != NRANK || (unsigned)rank >= (unsigned)NRANK || n_in != 2 || ws_bytes < WS_NEED ||
      in_sizes[0] != 4096 * 512 || in_sizes[1] != KK * NN || out_size != 512 * NN) {
    fprintf(stderr, "kernel_launch: unexpected problem shape\n");
    return;
  }
  static bool init = false;
  if (!init) {
    HIP_CHECK(hipFuncSetAttribute((const void*)a2a_gemm_q8, hipFuncAttributeMaxDynamicSharedMemorySize, LDS_BYTES));
    init = true;
  }
  Params P;
  P.x = (const char*)d_in[0];
  P.w = (const char*)d_in[1];
  P.out = (signed char*)d_out;
  for (int p = 0; p < NRANK; ++p) P.ws[p] = (char*)d_ws_all[p];
  P.rank = rank;
  hipLaunchKernelGGL(a2a_gemm_q8, dim3(NWG), dim3(NTHREADS), LDS_BYTES, stream, P);
  HIP_CHECK(hipGetLastError());
}
```

```cpp
#include <hip/hip_runtime.h>
#include <rccl/rccl.h>
#include <cstdio>
#include <type_traits>

#define HIP_CHECK(expr) do { hipError_t _e = (expr); if (_e != hipSuccess) { \
    fprintf(stderr, "HIP error %s at %s:%d\n", hipGetErrorString(_e), __FILE__, __LINE__); \
    return; } } while (0)

#define LAS __attribute__((address_space(3)))
#define CAS __attribute__((address_space(4)))
typedef short bf16x8 __attribute__((ext_vector_type(8)));
typedef short bf16x4 __attribute__((ext_vector_type(4)));
typedef float f32x4 __attribute__((ext_vector_type(4)));
typedef unsigned u32x4 __attribute__((ext_vector_type(4)));
typedef unsigned long long u64;

namespace {

constexpr int NRANK = 8;
constexpr int NN = 16384, KK = 4096;
constexpr int NT = KK / 64;
constexpr int A_ST = 32768, B_ST = 16384, NA = 3, NB = 4;
constexpr int B_OFF = 0, A_OFF = NB * B_ST, LDS_BYTES = NA * A_ST + NB * B_ST;
constexpr int NTHREADS = 768, NWG = 256;

constexpr size_t WS_SEQ = 0, WS_TMO = 128;
constexpr size_t WS_FLAGS = 4096;
constexpr size_t WS_LGRAN = 8192;
constexpr size_t WS_GGRAN = 16384;
constexpr int NREP = 32;
constexpr size_t WS_STAGE = 1u << 20;
constexpr size_t WS_NEED = WS_STAGE + (size_t)56 * 65536;

constexpr u64 DEADLINE_TICKS = 100000000ull;
#define PACE 10

struct Params {
  const char* x;
  const char* w;
  signed char* out;
  char* ws[NRANK];
  int rank;
};
#define STAMPC(i) do { } while (0)
#define STAMP(i) do { } while (0)

#define WAIT_V(n) asm volatile("s_waitcnt vmcnt(" #n ")" ::: "memory")
#define WAIT_L(n) asm volatile("s_waitcnt lgkmcnt(" #n ")" ::: "memory")
#define BAR __builtin_amdgcn_s_barrier()
#define SCHED __builtin_amdgcn_sched_barrier(0)
#define IC(x) std::integral_constant<int, (x)>{}

#define DEF_DMA4(NAME, POL) \
__device__ __forceinline__ void NAME(unsigned lds_addr, const char* gbase, unsigned v0, unsigned v1, unsigned v2, unsigned v3) { \
  asm volatile("s_mov_b32 m0, %0\n\ts_nop 0\n\t" \
               "global_load_lds_dwordx4 %2, %1" POL "\n\t" \
               "global_load_lds_dwordx4 %3, %1 offset:1024" POL "\n\t" \
               "global_load_lds_dwordx4 %4, %1 offset:2048" POL "\n\t" \
               "global_load_lds_dwordx4 %5, %1 offset:3072" POL \
               :: "s"(lds_addr), "s"(gbase), "v"(v0), "v"(v1), "v"(v2), "v"(v3) : "memory"); \
}
#define WPOL ""
DEF_DMA4(dma4, "")
DEF_DMA4(dma4_w, WPOL)
DEF_DMA4(dma4_sys, " sc0 sc1")

__device__ __forceinline__ unsigned sload_u32(const char* p) { return *(const CAS unsigned*)p; }
__device__ __forceinline__ bool vm_busy() { unsigned r; asm volatile("s_getreg_b32 %0, hwreg(HW_REG_IB_STS)" : "=s"(r) :: "memory"); return (r & 0x00c0000fu) != 0u; }
__device__ __forceinline__ u64 now() { return __builtin_amdgcn_s_memrealtime(); }
__device__ __forceinline__ bool timed_out(unsigned* tmo, unsigned seq) {
  return (unsigned)__builtin_amdgcn_readfirstlane(__hip_atomic_load(tmo, __ATOMIC_RELAXED, __HIP_MEMORY_SCOPE_AGENT)) == seq;
}
__device__ __forceinline__ void set_timeout(unsigned* tmo, unsigned seq) { __hip_atomic_store(tmo, seq, __ATOMIC_RELAXED, __HIP_MEMORY_SCOPE_AGENT); }

__device__ __forceinline__ unsigned wave_max_u32(unsigned v) {
  v = max(v, (unsigned)__builtin_amdgcn_update_dpp(0, (int)v, 0x111, 0xf, 0xf, true));
  v = max(v, (unsigned)__builtin_amdgcn_update_dpp(0, (int)v, 0x112, 0xf, 0xf, true));
  v = max(v, (unsigned)__builtin_amdgcn_update_dpp(0, (int)v, 0x114, 0xf, 0xf, true));
  v = max(v, (unsigned)__builtin_amdgcn_update_dpp(0, (int)v, 0x118, 0xf, 0xf, true));
  v = max(v, (unsigned)__builtin_amdgcn_update_dpp(0, (int)v, 0x142, 0xa, 0xf, true));
  v = max(v, (unsigned)__builtin_amdgcn_update_dpp(0, (int)v, 0x143, 0xc, 0xf, true));
  return (unsigned)__builtin_amdgcn_readlane((int)v, 63);
}

__device__ __forceinline__ int step_k(int t, int rank) {
  if (t < 8) return rank * 512 + t * 64;
  const int u = t - 8, j = (u * 37) >> 8, i = u - j * 7;
  return ((rank + 1 + i) & 7) * 512 + j * 64;
}

__global__ __launch_bounds__(NTHREADS) void a2a_gemm_q8(Params P) {
  extern __shared__ __attribute__((aligned(16))) unsigned char shm[];
  LAS unsigned char* lds = (LAS unsigned char*)shm;
  const unsigned ldsb = (unsigned)(size_t)lds;
  const int tid = threadIdx.x, wid = __builtin_amdgcn_readfirstlane(tid >> 6), lane = tid & 63;
  const int b = blockIdx.x, xcd = b & 7, idx = b >> 3;
  const int ct = xcd * 16 + (idx >> 1), rt = idx & 1;
  const int rank = P.rank;
  char* const myws = P.ws[rank];
  unsigned* const tmo = (unsigned*)(myws + WS_TMO);

  if (wid >= 8) {
    if (wid < 10) {
      const int a = wid - 8;
      unsigned vS[2], vL[4];
#pragma unroll
      for (int h = 0; h < 2; ++h) { const int f = (4 * h + (lane >> 4)) & 7; vS[h] = (unsigned)((lane >> 3) * 128 + (((lane & 7) ^ f) * 16)); }
#pragma unroll
      for (int i = 0; i < 4; ++i) { const int f = (4 * i + (lane >> 4)) & 7; vL[i] = (unsigned)((lane >> 3) * 1024 + (((lane & 7) ^ f) * 16) + i * (8192 - 1024)); }
      const char* xb = P.x + (size_t)(rank * 512 + rt * 256 + a * 128) * 1024;
      const char* sbp = myws + WS_STAGE + (size_t)rt * 32768 + a * 16384;
      const char* flags = myws + WS_FLAGS;
      const unsigned ldst = ldsb + A_OFF + a * 16384;
      const unsigned foff = (unsigned)(lane & 7) * 16;
      auto issue_half = [&](int t, int h) {
        const unsigned l = ldst + (t % NA) * A_ST + h * 8192;
        if (t < 8) { const char* g = xb + t * 128 + h * (64 * 1024);
          dma4(l, g, vL[0], vL[1], vL[2], vL[3]); dma4(l + 4096, g + 32 * 1024, vL[0], vL[1], vL[2], vL[3]); }
        else { const char* g = sbp + (size_t)(t - 8) * 65536 + h * 8192;
          dma4_sys(l, g, vS[0], vS[1], vS[0], vS[1]); dma4_sys(l + 4096, g + 4096, vS[0], vS[1], vS[0], vS[1]); }
      };
      issue_half(0, 0); issue_half(0, 1); issue_half(1, 0); issue_half(1, 1);
      const unsigned seq = sload_u32(myws + WS_SEQ) + 1u;
      WAIT_V(16); BAR;
      int cur_j = -1; unsigned mask = 0; bool dead = false;
      for (int t = 0; t < NT; ++t) {
        const int s = t + 2;
        if (s >= 8 && s < NT && !dead) {
          const int u = s - 8, j = (u * 37) >> 8, src = (rank + 1 + (u - j * 7)) & 7;
          if (j != cur_j || !((mask >> src) & 1u)) {
            cur_j = j;
            const u64 t0 = now();
            if (a == 0) STAMP(64 + (s - 8));
            for (;;) {
              u32x4 f;
              asm volatile("global_load_dwordx4 %0, %1, %2 sc0 sc1\n\ts_waitcnt vmcnt(0)" : "=&v"(f) : "v"(foff), "s"(flags + j * 128) : "memory");
              const bool ok = f[0] >= seq && f[1] >= seq && f[2] >= seq && f[3] >= seq;
              mask = (unsigned)__builtin_amdgcn_ballot_w64(ok) & 0xffu;
              if ((mask >> src) & 1u) break;
              if (timed_out(tmo, seq) || now() - t0 > DEADLINE_TICKS) { set_timeout(tmo, seq); dead = true; break; }
              __builtin_amdgcn_s_sleep(1);
            }
          }
        }
        bool poll = false; int j1 = 0;
        if (s + 1 >= 8 && s + 1 < NT && !dead) {
          const int u = s + 1 - 8; j1 = (u * 37) >> 8; const int src = (rank + 1 + (u - j1 * 7)) & 7;
          poll = (j1 != cur_j) || !((mask >> src) & 1u);
        }
        const bool live = s < NT && !dead;
        if (poll) {
          u32x4 f;
          asm volatile("global_load_dwordx4 %0, %1, %2 sc0 sc1" : "=&v"(f) : "v"(foff), "s"(flags + j1 * 128) : "memory");
          issue_half(s, 0); issue_half(s, 1);
          BAR;
          WAIT_V(16);
          asm volatile("" : "+v"(f));
          const bool ok = f[0] >= seq && f[1] >= seq && f[2] >= seq && f[3] >= seq;
          const unsigned nm = (unsigned)__builtin_amdgcn_ballot_w64(ok) & 0xffu;
          cur_j = j1; mask = nm;
          BAR;
        } else if (live) {
          issue_half(s, 0); issue_half(s, 1);
          BAR;
          WAIT_V(16);
          BAR;
        } else {
          BAR;
          WAIT_V(0);
          BAR;
        }
      }
      BAR;
    } else if (wid == 10) {
      unsigned vB[4];
#pragma unroll
      for (int i = 0; i < 4; ++i) { const int q = lane >> 4, s = (q << 1) | ((i >> 1) & 1), u = ((lane & 15) >> 1) ^ s;
        vB[i] = (unsigned)(q * (NN * 2) + (2 * u + (lane & 1)) * 16 + i * (4 * NN * 2 - 1024)); }
      const char* wb = P.w + (size_t)ct * 256;
      const unsigned ldst = ldsb + B_OFF;
      auto issue_half = [&](int t, int h) {
        const char* g = wb + (size_t)(step_k(t, rank) + h * 32) * (NN * 2);
        const unsigned l = ldst + (t % NB) * B_ST + h * 8192;
        dma4_w(l, g, vB[0], vB[1], vB[2], vB[3]); dma4_w(l + 4096, g + (size_t)16 * NN * 2, vB[0], vB[1], vB[2], vB[3]);
      };
      issue_half(0, 0); issue_half(0, 1); issue_half(1, 0); issue_half(1, 1); issue_half(2, 0); issue_half(2, 1);
      WAIT_V(32); BAR;
      for (int t = 0; t < NT - 3; ++t) {
        BAR;
        issue_half(t + 3, 0); issue_half(t + 3, 1);
        WAIT_V(32);
        BAR;
      }
      BAR; WAIT_V(16); BAR;
      BAR; WAIT_V(0); BAR;
      BAR; BAR;
      BAR;
    } else {
      if (b >= 224) return;
      const int j = b / 28, r = b - j * 28, i = r >> 2, q = r & 3;
      const int dst = (rank + 1 + i) & 7, u = 7 * j + (6 - i);
      const char* src = P.x + (size_t)(dst * 512 + q * 128) * 1024 + j * 128;
      char* dws = P.ws[dst];
      char* dbase = dws + WS_STAGE + (size_t)u * 65536 + q * 16384;
      const unsigned voff = (unsigned)((lane >> 3) * 1024 + (lane & 7) * 16);
      u32x4 d[16];
#pragma unroll
      for (int k = 0; k < 16; ++k)
        asm volatile("global_load_dwordx4 %0, %1, %2" : "=&v"(d[k]) : "v"(voff), "s"(src + k * 8192) : "memory");
      const unsigned seq = sload_u32(myws + WS_SEQ) + 1u;
      const int start = PACE * j;
      int s = 0;
      while (s < start || vm_busy()) { BAR; ++s; }
      WAIT_V(0);
      STAMP(32);
      const unsigned soff = (unsigned)lane * 16;
#pragma unroll
      for (int k = 0; k < 16; ++k)
        asm volatile("global_store_dwordx4 %0, %1, %2 sc0 sc1\n\ts_nop 1" :: "v"(soff), "v"(d[k]), "s"(dbase + k * 1024) : "memory");
      while (vm_busy()) BAR;
      WAIT_V(0);
      STAMP(34);
      if (lane == 0)
        __hip_atomic_store((unsigned*)(dws + WS_FLAGS + j * 128 + rank * 16 + q * 4), seq, __ATOMIC_RELAXED, __HIP_MEMORY_SCOPE_SYSTEM);
    }
    return;
  }

  const int grp = wid >> 2, wm = wid & 3, wn = grp;
  const int fr = lane & 15, fq = lane >> 4;
  unsigned aoff[2];
#pragma unroll
  for (int kh = 0; kh < 2; ++kh) aoff[kh] = ldsb + (unsigned)(A_OFF + (wm * 64 + fr) * 128 + (((4 * kh + fq) ^ ((fr >> 1) & 7)) * 16));
  unsigned boff[4];
  { const int q = (lane & 15) >> 2, p = lane & 3; const int s = (q << 1) | (fq & 1);
#pragma unroll
    for (int n = 0; n < 4; ++n) { const int u = (wn * 4 + n) ^ s; boff[n] = ldsb + (unsigned)(B_OFF + (8 * fq + q) * 256 + u * 32 + p * 8); } }
  unsigned aoffH[2] = {aoff[0] + 65536u, aoff[1] + 65536u};
  asm volatile("" : "+v"(aoffH[0]), "+v"(aoffH[1]));

  f32x4 acc[4][4];
#pragma unroll
  for (int m = 0; m < 4; ++m)
#pragma unroll
    for (int n = 0; n < 4; ++n) acc[m][n] = (f32x4){0.f, 0.f, 0.f, 0.f};
  bf16x8 af[4][2], bfr[4][2];
  const unsigned seq = sload_u32(myws + WS_SEQ) + 1u;

  if (wid == 0) STAMP(0);
  BAR;
  if (grp == 1) BAR;
  if (wid == 0) STAMP(1);

#define TRD(dst, addr, off) asm volatile("ds_read_b64_tr_b16 %0, %1 offset:%2" : "=&v"(dst) : "v"(addr), "i"(off) : "memory")
#define ARD(dst, addr, off) asm volatile("ds_read_b128 %0, %1 offset:%2" : "=&v"(dst) : "v"(addr), "i"(off) : "memory")
  auto step = [&](auto SAc, auto SBc) {
    constexpr int SA = decltype(SAc)::value, SB = decltype(SBc)::value;
    bf16x4 lo[4][2], hi[4][2];
#pragma unroll
    for (int n = 0; n < 4; ++n) {
      TRD(lo[n][0], boff[n], SB * B_ST); TRD(hi[n][0], boff[n], SB * B_ST + 1024);
      TRD(lo[n][1], boff[n], SB * B_ST + 8192); TRD(hi[n][1], boff[n], SB * B_ST + 9216);
    }
#pragma unroll
    for (int m = 0; m < 4; ++m) {
      if constexpr (SA < 2) { ARD(af[m][0], aoff[0], SA * A_ST + m * 2048); ARD(af[m][1], aoff[1], SA * A_ST + m * 2048); }
      else { ARD(af[m][0], aoffH[0], m * 2048); ARD(af[m][1], aoffH[1], m * 2048); }
    }
    WAIT_L(0);
    SCHED;
#pragma unroll
    for (int n = 0; n < 4; ++n)
#pragma unroll
      for (int kh = 0; kh < 2; ++kh) bfr[n][kh] = __builtin_shufflevector(lo[n][kh], hi[n][kh], 0, 1, 2, 3, 4, 5, 6, 7);
    BAR;
    __builtin_amdgcn_s_setprio(1);
#pragma unroll
    for (int kh = 0; kh < 2; ++kh)
#pragma unroll
      for (int m = 0; m < 4; ++m)
#pragma unroll
        for (int n = 0; n < 4; ++n) acc[m][n] = __builtin_amdgcn_mfma_f32_16x16x32_bf16(bfr[n][kh], af[m][kh], acc[m][n], 0, 0, 0);
    __builtin_amdgcn_s_setprio(0);
    BAR;
    SCHED;
  };
  for (int it = 0; it < 5; ++it) {
    step(IC(0), IC(0)); step(IC(1), IC(1)); step(IC(2), IC(2)); step(IC(0), IC(3));
    if (wid == 0) { STAMP(8 + 3 * it); STAMPC(40 + it); }
    step(IC(1), IC(0)); step(IC(2), IC(1)); step(IC(0), IC(2)); step(IC(1), IC(3));
    if (wid == 0) STAMP(9 + 3 * it);
    step(IC(2), IC(0)); step(IC(0), IC(1)); step(IC(1), IC(2)); step(IC(2), IC(3));
    if (wid == 0) STAMP(10 + 3 * it);
  }
  step(IC(0), IC(0)); step(IC(1), IC(1)); step(IC(2), IC(2)); step(IC(0), IC(3));

  if (wid == 0) STAMP(2);
  float mx = 0.f;
#pragma unroll
  for (int m = 0; m < 4; ++m)
#pragma unroll
    for (int n = 0; n < 4; ++n) {
      mx = __builtin_fmaxf(mx, __builtin_fmaxf(__builtin_fabsf(acc[m][n][0]), __builtin_fabsf(acc[m][n][1])));
      mx = __builtin_fmaxf(mx, __builtin_fmaxf(__builtin_fabsf(acc[m][n][2]), __builtin_fabsf(acc[m][n][3])));
    }
  const unsigned wmx = wave_max_u32(__float_as_uint(mx));
  if (grp == 0) BAR;
  LAS unsigned* red = (LAS unsigned*)lds;
  if (lane == 0) red[wid] = wmx;
  WAIT_L(0); BAR;

  if (wid == 0) {
    unsigned v = red[lane & 7];
    v = max(v, (unsigned)__builtin_amdgcn_update_dpp(0, (int)v, 0x111, 0xf, 0xf, true));
    v = max(v, (unsigned)__builtin_amdgcn_update_dpp(0, (int)v, 0x112, 0xf, 0xf, true));
    v = max(v, (unsigned)__builtin_amdgcn_update_dpp(0, (int)v, 0x114, 0xf, 0xf, true));
    const unsigned gmx = (unsigned)__builtin_amdgcn_readlane((int)v, 7);
    u64* lg = (u64*)(myws + WS_LGRAN);
    if (lane == 0) __hip_atomic_store(lg + b, ((u64)seq << 32) | gmx, __ATOMIC_RELAXED, __HIP_MEMORY_SCOPE_AGENT);
    STAMP(3);
    bool dead = false;
    if (b < NRANK) {
      const u64 t0 = now(); unsigned am = 0;
      for (unsigned it = 1;; ++it) {
        bool ok = true; am = 0;
#pragma unroll
        for (int k = 0; k < 4; ++k) { const u64 g = __hip_atomic_load(lg + k * 64 + lane, __ATOMIC_RELAXED, __HIP_MEMORY_SCOPE_AGENT);
          ok = ok && (unsigned)(g >> 32) >= seq; am = max(am, (unsigned)g); }
        if (__builtin_amdgcn_ballot_w64(ok) == ~0ull) break;
        if ((it & 15u) == 0u && (timed_out(tmo, seq) || now() - t0 > DEADLINE_TICKS)) { set_timeout(tmo, seq); dead = true; break; }
      }
      am = wave_max_u32(am);
      if (!dead && lane < NREP)
        __hip_atomic_store((u64*)(P.ws[b] + WS_GGRAN + lane * 128 + rank * 8), ((u64)seq << 32) | am, __ATOMIC_RELAXED, __HIP_MEMORY_SCOPE_SYSTEM);
    }
    STAMP(4);
    const u64* gg = (const u64*)(myws + WS_GGRAN + (b & (NREP - 1)) * 128) + (lane & 7);
    const u64 t0 = now(); unsigned am = 0;
    for (unsigned it = 1; !dead; ++it) {
      const u64 g = __hip_atomic_load(gg, __ATOMIC_RELAXED, __HIP_MEMORY_SCOPE_SYSTEM);
      am = (unsigned)g;
      if (__builtin_amdgcn_ballot_w64((unsigned)(g >> 32) >= seq) == ~0ull) break;
      if ((it & 15u) == 0u && (timed_out(tmo, seq) || now() - t0 > DEADLINE_TICKS)) { set_timeout(tmo, seq); dead = true; break; }
      __builtin_amdgcn_s_sleep(1);
    }
    am = max(am, (unsigned)__builtin_amdgcn_update_dpp(0, (int)am, 0x111, 0xf, 0xf, true));
    am = max(am, (unsigned)__builtin_amdgcn_update_dpp(0, (int)am, 0x112, 0xf, 0xf, true));
    am = max(am, (unsigned)__builtin_amdgcn_update_dpp(0, (int)am, 0x114, 0xf, 0xf, true));
    am = (unsigned)__builtin_amdgcn_readlane((int)am, 7);
    STAMP(5);
    const bool bad = dead || timed_out(tmo, seq);
    if (lane == 0) { red[8] = am; red[9] = bad ? 1u : 0u;
      if (b == 0) __hip_atomic_store((unsigned*)(myws + WS_SEQ), seq, __ATOMIC_RELAXED, __HIP_MEMORY_SCOPE_AGENT); }
  }
  WAIT_L(0); BAR;
  const float amax = __uint_as_float(red[8]);
  const bool bad = red[9] != 0u;
  const float inv = amax > 0.f ? 1.0f / (amax / 127.0f) : 0.f;

  signed char* orow = P.out + (size_t)(rt * 256 + wm * 64 + fr) * NN + ct * 128 + wn * 64 + fq * 16;
#pragma unroll
  for (int m = 0; m < 4; ++m) {
    unsigned d[4];
#pragma unroll
    for (int n = 0; n < 4; ++n) {
      unsigned q[4];
#pragma unroll
      for (int e = 0; e < 4; ++e) q[e] = __float_as_uint(__builtin_fmaf(acc[m][n][e], inv, 12582912.0f));
      const unsigned p01 = __builtin_amdgcn_perm(q[1], q[0], 0x0c0c0400u), p23 = __builtin_amdgcn_perm(q[3], q[2], 0x04000c0cu);
      d[n] = p01 | p23;
    }
    auto r0 = __builtin_amdgcn_permlane32_swap(d[0], d[2], false, false); d[0] = r0[0]; d[2] = r0[1];
    auto r1 = __builtin_amdgcn_permlane32_swap(d[1], d[3], false, false); d[1] = r1[0]; d[3] = r1[1];
    auto r2 = __builtin_amdgcn_permlane16_swap(d[0], d[1], false, false); d[0] = r2[0]; d[1] = r2[1];
    auto r3 = __builtin_amdgcn_permlane16_swap(d[2], d[3], false, false); d[2] = r3[0]; d[3] = r3[1];
    u32x4 o = (u32x4){d[0], d[1], d[2], d[3]};
    if (bad) o = (u32x4){0x80808080u, 0x80808080u, 0x80808080u, 0x80808080u};
    *(u32x4*)(orow + (size_t)m * 16 * NN) = o;
  }
  if (wid == 0) STAMP(6);
}

}

extern "C" void kernel_launch(int rank, int nranks, ncclComm_t comm,
                              hipStream_t stream,
                              void* const* d_in, const int* in_sizes, int n_in,
                              void* d_out, int out_size,
                              const int* in_dtypes, int out_dtype,
                              void* const* d_out_all,
                              void* const* d_ws_all, size_t ws_bytes) {
  (void)comm; (void)in_dtypes; (void)out_dtype; (void)d_out_all;
  if (nranks != NRANK || (unsigned)rank >= (unsigned)NRANK || n_in != 2 || ws_bytes < WS_NEED ||
      in_sizes[0] != 4096 * 512 || in_sizes[1] != KK * NN || out_size != 512 * NN) {
    fprintf(stderr, "kernel_launch: unexpected problem shape\n");
    return;
  }
  static bool init = false;
  if (!init) {
    HIP_CHECK(hipFuncSetAttribute((const void*)a2a_gemm_q8, hipFuncAttributeMaxDynamicSharedMemorySize, LDS_BYTES));
    init = true;
  }
  Params P;
  P.x = (const char*)d_in[0];
  P.w = (const char*)d_in[1];
  P.out = (signed char*)d_out;
  for (int p = 0; p < NRANK; ++p) P.ws[p] = (char*)d_ws_all[p];
  P.rank = rank;
  hipLaunchKernelGGL(a2a_gemm_q8, dim3(NWG), dim3(NTHREADS), LDS_BYTES, stream, P);
  HIP_CHECK(hipGetLastError());
}
```

```cpp
#include <hip/hip_runtime.h>
#include <rccl/rccl.h>
#include <cstdio>
#include <type_traits>

#define HIP_CHECK(expr) do { hipError_t _e = (expr); if (_e != hipSuccess) { \
    fprintf(stderr, "HIP error %s at %s:%d\n", hipGetErrorString(_e), __FILE__, __LINE__); \
    return; } } while (0)

#define LAS __attribute__((address_space(3)))
#define CAS __attribute__((address_space(4)))
typedef short bf16x8 __attribute__((ext_vector_type(8)));
typedef short bf16x4 __attribute__((ext_vector_type(4)));
typedef float f32x4 __attribute__((ext_vector_type(4)));
typedef unsigned u32x4 __attribute__((ext_vector_type(4)));
typedef unsigned long long u64;

namespace {

constexpr int NRANK = 8;
constexpr int NN = 16384, KK = 4096;
constexpr int NT = KK / 64;
constexpr int A_ST = 32768, B_ST = 16384, NA = 3, NB = 4;
constexpr int B_OFF = 0, A_OFF = NB * B_ST, LDS_BYTES = NA * A_ST + NB * B_ST;
constexpr int NTHREADS = 768, NWG = 256;

constexpr size_t WS_SEQ = 0, WS_TMO = 128;
constexpr size_t WS_FLAGS = 4096;
constexpr size_t WS_LGRAN = 8192;
constexpr size_t WS_GGRAN = 16384;
constexpr int NREP = 32;
constexpr size_t WS_STAGE = 1u << 20;
constexpr size_t WS_NEED = WS_STAGE + (size_t)56 * 65536;

#define SWAPK 0
#define PFA 0
#define PFA_FIRST 1
#define PFA_LAG 12
#define PFA_SLEEP 4
constexpr u64 DEADLINE_TICKS = 100000000ull;
#define PACE 10

struct Params {
  const char* x;
  const char* w;
  signed char* out;
  char* ws[NRANK];
  int rank;
};
#define STAMP3(i) do { } while (0)
#define STAMP2(i) do { } while (0)
#define STAMPC2(i) do { } while (0)
#define STAMPC(i) do { } while (0)
#define STAMP(i) do { } while (0)

#define WAIT_V(n) asm volatile("s_waitcnt vmcnt(" #n ")" ::: "memory")
#define WAIT_L(n) asm volatile("s_waitcnt lgkmcnt(" #n ")" ::: "memory")
#define BAR __builtin_amdgcn_s_barrier()
#define SCHED __builtin_amdgcn_sched_barrier(0)
#define IC(x) std::integral_constant<int, (x)>{}

#define DEF_DMA4(NAME, POL) \
__device__ __forceinline__ void NAME(unsigned lds_addr, const char* gbase, unsigned v0, unsigned v1, unsigned v2, unsigned v3) { \
  asm volatile("s_mov_b32 m0, %0\n\ts_nop 0\n\t" \
               "global_load_lds_dwordx4 %2, %1" POL "\n\t" \
               "global_load_lds_dwordx4 %3, %1 offset:1024" POL "\n\t" \
               "global_load_lds_dwordx4 %4, %1 offset:2048" POL "\n\t" \
               "global_load_lds_dwordx4 %5, %1 offset:3072" POL \
               :: "s"(lds_addr), "s"(gbase), "v"(v0), "v"(v1), "v"(v2), "v"(v3) : "memory"); \
}
#define WPOL " nt"
DEF_DMA4(dma4, "")
DEF_DMA4(dma4_w, WPOL)
DEF_DMA4(dma4_sys, " sc0 sc1")

__device__ __forceinline__ unsigned sload_u32(const char* p) { return *(const CAS unsigned*)p; }
__device__ __forceinline__ bool vm_busy() { unsigned r; asm volatile("s_getreg_b32 %0, hwreg(HW_REG_IB_STS)" : "=s"(r) :: "memory"); return (r & 0x00c0000fu) != 0u; }
__device__ __forceinline__ u64 now() { return __builtin_amdgcn_s_memrealtime(); }
__device__ __forceinline__ bool timed_out(unsigned* tmo, unsigned seq) {
  return (unsigned)__builtin_amdgcn_readfirstlane(__hip_atomic_load(tmo, __ATOMIC_RELAXED, __HIP_MEMORY_SCOPE_AGENT)) == seq;
}
__device__ __forceinline__ void set_timeout(unsigned* tmo, unsigned seq) { __hip_atomic_store(tmo, seq, __ATOMIC_RELAXED, __HIP_MEMORY_SCOPE_AGENT); }

__device__ __forceinline__ unsigned wave_max_u32(unsigned v) {
  v = max(v, (unsigned)__builtin_amdgcn_update_dpp(0, (int)v, 0x111, 0xf, 0xf, true));
  v = max(v, (unsigned)__builtin_amdgcn_update_dpp(0, (int)v, 0x112, 0xf, 0xf, true));
  v = max(v, (unsigned)__builtin_amdgcn_update_dpp(0, (int)v, 0x114, 0xf, 0xf, true));
  v = max(v, (unsigned)__builtin_amdgcn_update_dpp(0, (int)v, 0x118, 0xf, 0xf, true));
  v = max(v, (unsigned)__builtin_amdgcn_update_dpp(0, (int)v, 0x142, 0xa, 0xf, true));
  v = max(v, (unsigned)__builtin_amdgcn_update_dpp(0, (int)v, 0x143, 0xc, 0xf, true));
  return (unsigned)__builtin_amdgcn_readlane((int)v, 63);
}

__device__ __forceinline__ int step_k(int t, int rank) {
  if (t < 8) return rank * 512 + t * 64;
  const int u = t - 8, j = (u * 37) >> 8, i = u - j * 7;
  return ((rank + 1 + i) & 7) * 512 + j * 64;
}

__global__ __launch_bounds__(NTHREADS) void a2a_gemm_q8(Params P) {
  extern __shared__ __attribute__((aligned(16))) unsigned char shm[];
  LAS unsigned char* lds = (LAS unsigned char*)shm;
  const unsigned ldsb = (unsigned)(size_t)lds;
  const int tid = threadIdx.x, wid = __builtin_amdgcn_readfirstlane(tid >> 6), lane = tid & 63;
  const int b = blockIdx.x, xcd = b & 7, idx = b >> 3;
  const int ct = xcd * 16 + (idx >> 1), rt = idx & 1;
  const int rank = P.rank;
  const int sw = rt & SWAPK;
  char* const myws = P.ws[rank];
  unsigned* const tmo = (unsigned*)(myws + WS_TMO);

  if (wid >= 8) {
    if (wid < 10) {
      const int a = wid - 8;
      unsigned vS[2], vL[4];
#pragma unroll
      for (int h = 0; h < 2; ++h) { const int f = (4 * h + (lane >> 4)) & 7; vS[h] = (unsigned)((lane >> 3) * 128 + (((lane & 7) ^ f) * 16)); }
#pragma unroll
      for (int i = 0; i < 4; ++i) { const int f = (4 * i + (lane >> 4)) & 7; vL[i] = (unsigned)((lane >> 3) * 1024 + (((lane & 7) ^ f) * 16) + i * (8192 - 1024)); }
      const char* xb = P.x + (size_t)(rank * 512 + rt * 256 + a * 128) * 1024;
      const char* sbp = myws + WS_STAGE + (size_t)rt * 32768 + a * 16384;
      const char* flags = myws + WS_FLAGS;
      const unsigned ldst = ldsb + A_OFF + a * 16384;
      const unsigned foff = (unsigned)(lane & 7) * 16;
      auto issue_own = [&](unsigned l, const char* g) {
        dma4(l, g, vL[0], vL[1], vL[2], vL[3]); dma4(l + 4096, g + 32 * 1024, vL[0], vL[1], vL[2], vL[3]);
        dma4(l + 8192, g + 64 * 1024, vL[0], vL[1], vL[2], vL[3]); dma4(l + 12288, g + 96 * 1024, vL[0], vL[1], vL[2], vL[3]);
      };
      auto issue_peer = [&](unsigned l, const char* g) {
        dma4_sys(l, g, vS[0], vS[1], vS[0], vS[1]); dma4_sys(l + 4096, g + 4096, vS[0], vS[1], vS[0], vS[1]);
        dma4_sys(l + 8192, g + 8192, vS[0], vS[1], vS[0], vS[1]); dma4_sys(l + 12288, g + 12288, vS[0], vS[1], vS[0], vS[1]);
      };
      issue_own(ldst, xb + sw * 128); issue_own(ldst + A_ST, xb + (1 ^ sw) * 128);
      const unsigned seq = sload_u32(myws + WS_SEQ) + 1u;
      u64 ready = 0; bool dead = false;
      unsigned l_nx = ldst + 2 * A_ST; const char* g_nx = xb + (2 ^ sw) * 128; bool own_nx = true, poll_nx = false; int j_nx = 0;
      WAIT_V(16); BAR;
      for (int t = 0; t < NT; ++t) {
        const bool live = t + 2 < NT && !dead;
        if (live && poll_nx) {
          u32x4 f;
          asm volatile("global_load_dwordx4 %0, %1, %2 sc0 sc1" : "=&v"(f) : "v"(foff), "s"(flags + j_nx * 128) : "memory");
          if (own_nx) issue_own(l_nx, g_nx); else issue_peer(l_nx, g_nx);
          BAR;
          WAIT_V(16);
          asm volatile("" : "+v"(f));
          const bool ok = f[0] >= seq && f[1] >= seq && f[2] >= seq && f[3] >= seq;
          ready |= (u64)((unsigned)__builtin_amdgcn_ballot_w64(ok) & 0xffu) << (8 * j_nx);
        } else if (live) {
          if (own_nx) issue_own(l_nx, g_nx); else issue_peer(l_nx, g_nx);
          BAR;
          WAIT_V(16);
        } else {
          BAR;
          WAIT_V(0);
        }
        const int s = t + 3;
        if (s < NT && !dead) {
          const int k = s ^ sw;
          l_nx = l_nx == ldst + 2 * A_ST ? ldst : l_nx + A_ST;
          own_nx = k < 8;
          if (own_nx) g_nx = xb + k * 128;
          else {
            const int u = k - 8, j = (u * 37) >> 8, src = (rank + 1 + (u - j * 7)) & 7;
            g_nx = sbp + (size_t)u * 65536;
            if (!((ready >> (8 * j + src)) & 1ull)) {
              const u64 t0 = now();
              if (a == 0) STAMP3(64 + (s - 8));
              for (;;) {
                u32x4 f;
                asm volatile("global_load_dwordx4 %0, %1, %2 sc0 sc1\n\ts_waitcnt vmcnt(0)" : "=&v"(f) : "v"(foff), "s"(flags + j * 128) : "memory");
                const bool ok = f[0] >= seq && f[1] >= seq && f[2] >= seq && f[3] >= seq;
                ready |= (u64)((unsigned)__builtin_amdgcn_ballot_w64(ok) & 0xffu) << (8 * j);
                if ((ready >> (8 * j + src)) & 1ull) break;
                if (timed_out(tmo, seq) || now() - t0 > DEADLINE_TICKS) { set_timeout(tmo, seq); dead = true; break; }
                __builtin_amdgcn_s_sleep(1);
              }
            }
          }
          poll_nx = false;
          if (s + 1 >= 8 && s + 1 < NT) {
            const int u = ((s + 1) ^ sw) - 8; j_nx = (u * 37) >> 8; const int src = (rank + 1 + (u - j_nx * 7)) & 7;
            poll_nx = !((ready >> (8 * j_nx + src)) & 1ull);
          }
        }
        BAR;
      }
      BAR;
    } else if (wid == 10) {
      unsigned vB[4];
#pragma unroll
      for (int i = 0; i < 4; ++i) { const int q = lane >> 4, s = (q << 1) | ((i >> 1) & 1), u = ((lane & 15) >> 1) ^ s;
        vB[i] = (unsigned)(q * (NN * 2) + (2 * u + (lane & 1)) * 16 + i * (4 * NN * 2 - 1024)); }
      const char* wb = P.w + (size_t)ct * 256;
      const unsigned ldst = ldsb + B_OFF;
      auto issue = [&](int t) {
        const char* g = wb + (size_t)step_k(t ^ sw, rank) * (NN * 2);
        const unsigned l = ldst + (t % NB) * B_ST;
#pragma unroll
        for (int q = 0; q < 4; ++q) dma4_w(l + q * 4096, g + (size_t)q * 16 * NN * 2, vB[0], vB[1], vB[2], vB[3]);
      };
      issue(0); issue(1); issue(2);
      WAIT_V(32); BAR;
      for (int t = 0; t < NT - 3; ++t) {
        BAR;
        issue(t + 3);
        WAIT_V(32);
        BAR;
      }
      BAR; WAIT_V(16); BAR;
      BAR; WAIT_V(0); BAR;
      BAR; BAR;
      BAR;
    } else {
      const bool has_push = b < 224;
      const int j = has_push ? b / 28 : 8, r = b - j * 28, i = (r >> 2) & 7, q = r & 3;
      const int dst = (rank + 1 + i) & 7;
      u32x4 d[16];
      if (has_push) {
        const char* src = P.x + (size_t)(dst * 512 + q * 128) * 1024 + j * 128;
        const unsigned voff = (unsigned)((lane >> 3) * 1024 + (lane & 7) * 16);
#pragma unroll
        for (int k = 0; k < 16; ++k)
          asm volatile("global_load_dwordx4 %0, %1, %2" : "=&v"(d[k]) : "v"(voff), "s"(src + k * 8192) : "memory");
      }
      const unsigned seq = sload_u32(myws + WS_SEQ) + 1u;
      const u64 t0 = now();
      int s = 0;
      bool pushed = !has_push;
      auto try_push = [&]() {
        if (pushed || s < PACE * j) return;
        char* dws = P.ws[dst];
        char* dbase = dws + WS_STAGE + (size_t)(7 * j + (6 - i)) * 65536 + q * 16384;
        STAMP2(32);
        const unsigned soff = (unsigned)lane * 16;
#pragma unroll
        for (int k = 0; k < 16; ++k)
          asm volatile("global_store_dwordx4 %0, %1, %2 sc0 sc1\n\ts_nop 1" :: "v"(soff), "v"(d[k]), "s"(dbase + k * 1024) : "memory");
        while (vm_busy()) { BAR; ++s; }
        WAIT_V(0);
        STAMP2(34);
        if (lane == 0)
          __hip_atomic_store((unsigned*)(dws + WS_FLAGS + j * 128 + rank * 16 + q * 4), seq, __ATOMIC_RELAXED, __HIP_MEMORY_SCOPE_SYSTEM);
        pushed = true;
        while (vm_busy()) { BAR; ++s; }
      };
      while (vm_busy()) { BAR; ++s; }
      WAIT_V(0);
      try_push();
      while (!pushed) { BAR; ++s; try_push(); }
    }
    return;
  }

  const int grp = wid >> 2, wm = wid & 3, wn = grp;
  const int fr = lane & 15, fq = lane >> 4;
  unsigned aoff[2];
#pragma unroll
  for (int kh = 0; kh < 2; ++kh) aoff[kh] = ldsb + (unsigned)(A_OFF + (wm * 64 + fr) * 128 + (((4 * kh + fq) ^ ((fr >> 1) & 7)) * 16));
  unsigned boff[4];
  { const int q = (lane & 15) >> 2, p = lane & 3; const int s = (q << 1) | (fq & 1);
#pragma unroll
    for (int n = 0; n < 4; ++n) { const int u = (wn * 4 + n) ^ s; boff[n] = ldsb + (unsigned)(B_OFF + (8 * fq + q) * 256 + u * 32 + p * 8); } }
  unsigned aoffH[2] = {aoff[0] + 65536u, aoff[1] + 65536u};
  asm volatile("" : "+v"(aoffH[0]), "+v"(aoffH[1]));

  f32x4 acc[4][4];
#pragma unroll
  for (int m = 0; m < 4; ++m)
#pragma unroll
    for (int n = 0; n < 4; ++n) acc[m][n] = (f32x4){0.f, 0.f, 0.f, 0.f};
  bf16x8 af[4][2], bfr[4][2];
  const unsigned seq = sload_u32(myws + WS_SEQ) + 1u;

  if (wid == 0) STAMP(0);
  BAR;
  if (grp == 1) BAR;
  if (wid == 0) STAMP(1);

#define TRD(dst, addr, off) asm volatile("ds_read_b64_tr_b16 %0, %1 offset:%2" : "=&v"(dst) : "v"(addr), "i"(off) : "memory")
#define ARD(dst, addr, off) asm volatile("ds_read_b128 %0, %1 offset:%2" : "=&v"(dst) : "v"(addr), "i"(off) : "memory")
  auto step = [&](auto SAc, auto SBc) {
    constexpr int SA = decltype(SAc)::value, SB = decltype(SBc)::value;
    bf16x4 lo[4][2], hi[4][2];
#pragma unroll
    for (int n = 0; n < 4; ++n) {
      TRD(lo[n][0], boff[n], SB * B_ST); TRD(hi[n][0], boff[n], SB * B_ST + 1024);
      TRD(lo[n][1], boff[n], SB * B_ST + 8192); TRD(hi[n][1], boff[n], SB * B_ST + 9216);
    }
#pragma unroll
    for (int m = 0; m < 4; ++m) {
      if constexpr (SA < 2) { ARD(af[m][0], aoff[0], SA * A_ST + m * 2048); ARD(af[m][1], aoff[1], SA * A_ST + m * 2048); }
      else { ARD(af[m][0], aoffH[0], m * 2048); ARD(af[m][1], aoffH[1], m * 2048); }
    }
    WAIT_L(0);
    SCHED;
#pragma unroll
    for (int n = 0; n < 4; ++n)
#pragma unroll
      for (int kh = 0; kh < 2; ++kh) bfr[n][kh] = __builtin_shufflevector(lo[n][kh], hi[n][kh], 0, 1, 2, 3, 4, 5, 6, 7);
    BAR;
    __builtin_amdgcn_s_setprio(1);
#pragma unroll
    for (int kh = 0; kh < 2; ++kh)
#pragma unroll
      for (int m = 0; m < 4; ++m)
#pragma unroll
        for (int n = 0; n < 4; ++n) acc[m][n] = __builtin_amdgcn_mfma_f32_16x16x32_bf16(bfr[n][kh], af[m][kh], acc[m][n], 0, 0, 0);
    __builtin_amdgcn_s_setprio(0);
    BAR;
    SCHED;
  };
  for (int it = 0; it < 5; ++it) {
    step(IC(0), IC(0)); step(IC(1), IC(1)); step(IC(2), IC(2)); step(IC(0), IC(3));
    if (wid == 0) { STAMP3(8 + 3 * it); STAMPC2(40 + it); }
    step(IC(1), IC(0)); step(IC(2), IC(1)); step(IC(0), IC(2)); step(IC(1), IC(3));
    if (wid == 0) STAMP3(9 + 3 * it);
    step(IC(2), IC(0)); step(IC(0), IC(1)); step(IC(1), IC(2)); step(IC(2), IC(3));
    if (wid == 0) STAMP3(10 + 3 * it);
  }
  step(IC(0), IC(0)); step(IC(1), IC(1)); step(IC(2), IC(2)); step(IC(0), IC(3));

  if (wid == 0) STAMP(2);
  float mx = 0.f;
#pragma unroll
  for (int m = 0; m < 4; ++m)
#pragma unroll
    for (int n = 0; n < 4; ++n) {
      mx = __builtin_fmaxf(mx, __builtin_fmaxf(__builtin_fabsf(acc[m][n][0]), __builtin_fabsf(acc[m][n][1])));
      mx = __builtin_fmaxf(mx, __builtin_fmaxf(__builtin_fabsf(acc[m][n][2]), __builtin_fabsf(acc[m][n][3])));
    }
  const unsigned wmx = wave_max_u32(__float_as_uint(mx));
  if (grp == 0) BAR;
  LAS unsigned* red = (LAS unsigned*)lds;
  if (lane == 0) red[wid] = wmx;
  WAIT_L(0); BAR;

  if (wid == 0) {
    unsigned v = red[lane & 7];
    v = max(v, (unsigned)__builtin_amdgcn_update_dpp(0, (int)v, 0x111, 0xf, 0xf, true));
    v = max(v, (unsigned)__builtin_amdgcn_update_dpp(0, (int)v, 0x112, 0xf, 0xf, true));
    v = max(v, (unsigned)__builtin_amdgcn_update_dpp(0, (int)v, 0x114, 0xf, 0xf, true));
    const unsigned gmx = (unsigned)__builtin_amdgcn_readlane((int)v, 7);
    u64* lg = (u64*)(myws + WS_LGRAN);
    if (lane == 0) __hip_atomic_store(lg + b, ((u64)seq << 32) | gmx, __ATOMIC_RELAXED, __HIP_MEMORY_SCOPE_AGENT);
    STAMP(3);
    bool dead = false;
    if (b < NRANK) {
      const u64 t0 = now(); unsigned am = 0;
      for (unsigned it = 1;; ++it) {
        bool ok = true; am = 0;
#pragma unroll
        for (int k = 0; k < 4; ++k) { const u64 g = __hip_atomic_load(lg + k * 64 + lane, __ATOMIC_RELAXED, __HIP_MEMORY_SCOPE_AGENT);
          ok = ok && (unsigned)(g >> 32) >= seq; am = max(am, (unsigned)g); }
        if (__builtin_amdgcn_ballot_w64(ok) == ~0ull) break;
        if ((it & 15u) == 0u && (timed_out(tmo, seq) || now() - t0 > DEADLINE_TICKS)) { set_timeout(tmo, seq); dead = true; break; }
      }
      am = wave_max_u32(am);
      if (!dead && lane < NREP)
        __hip_atomic_store((u64*)(P.ws[b] + WS_GGRAN + lane * 128 + rank * 8), ((u64)seq << 32) | am, __ATOMIC_RELAXED, __HIP_MEMORY_SCOPE_SYSTEM);
    }
    STAMP(4);
    const u64* gg = (const u64*)(myws + WS_GGRAN + (b & (NREP - 1)) * 128) + (lane & 7);
    const u64 t0 = now(); unsigned am = 0;
    for (unsigned it = 1; !dead; ++it) {
      const u64 g = __hip_atomic_load(gg, __ATOMIC_RELAXED, __HIP_MEMORY_SCOPE_SYSTEM);
      am = (unsigned)g;
      if (__builtin_amdgcn_ballot_w64((unsigned)(g >> 32) >= seq) == ~0ull) break;
      if ((it & 15u) == 0u && (timed_out(tmo, seq) || now() - t0 > DEADLINE_TICKS)) { set_timeout(tmo, seq); dead = true; break; }
      __builtin_amdgcn_s_sleep(1);
    }
    am = max(am, (unsigned)__builtin_amdgcn_update_dpp(0, (int)am, 0x111, 0xf, 0xf, true));
    am = max(am, (unsigned)__builtin_amdgcn_update_dpp(0, (int)am, 0x112, 0xf, 0xf, true));
    am = max(am, (unsigned)__builtin_amdgcn_update_dpp(0, (int)am, 0x114, 0xf, 0xf, true));
    am = (unsigned)__builtin_amdgcn_readlane((int)am, 7);
    STAMP(5);
    const bool bad = dead || timed_out(tmo, seq);
    if (lane == 0) { red[8] = am; red[9] = bad ? 1u : 0u;
      if (b == 0) __hip_atomic_store((unsigned*)(myws + WS_SEQ), seq, __ATOMIC_RELAXED, __HIP_MEMORY_SCOPE_AGENT); }
  }
  WAIT_L(0); BAR;
  const float amax = __uint_as_float(red[8]);
  const bool bad = red[9] != 0u;
  const float inv = amax > 0.f ? 1.0f / (amax / 127.0f) : 0.f;

  signed char* orow = P.out + (size_t)(rt * 256 + wm * 64 + fr) * NN + ct * 128 + wn * 64 + fq * 16;
#pragma unroll
  for (int m = 0; m < 4; ++m) {
    unsigned d[4];
#pragma unroll
    for (int n = 0; n < 4; ++n) {
      unsigned q[4];
#pragma unroll
      for (int e = 0; e < 4; ++e) q[e] = __float_as_uint(__builtin_fmaf(acc[m][n][e], inv, 12582912.0f));
      const unsigned p01 = __builtin_amdgcn_perm(q[1], q[0], 0x0c0c0400u), p23 = __builtin_amdgcn_perm(q[3], q[2], 0x04000c0cu);
      d[n] = p01 | p23;
    }
    auto r0 = __builtin_amdgcn_permlane32_swap(d[0], d[2], false, false); d[0] = r0[0]; d[2] = r0[1];
    auto r1 = __builtin_amdgcn_permlane32_swap(d[1], d[3], false, false); d[1] = r1[0]; d[3] = r1[1];
    auto r2 = __builtin_amdgcn_permlane16_swap(d[0], d[1], false, false); d[0] = r2[0]; d[1] = r2[1];
    auto r3 = __builtin_amdgcn_permlane16_swap(d[2], d[3], false, false); d[2] = r3[0]; d[3] = r3[1];
    u32x4 o = (u32x4){d[0], d[1], d[2], d[3]};
    if (bad) o = (u32x4){0x80808080u, 0x80808080u, 0x80808080u, 0x80808080u};
    *(u32x4*)(orow + (size_t)m * 16 * NN) = o;
  }
  if (wid == 0) STAMP(6);
}

}

extern "C" void kernel_launch(int rank, int nranks, ncclComm_t comm,
                              hipStream_t stream,
                              void* const* d_in, const int* in_sizes, int n_in,
                              void* d_out, int out_size,
                              const int* in_dtypes, int out_dtype,
                              void* const* d_out_all,
                              void* const* d_ws_all, size_t ws_bytes) {
  (void)comm; (void)in_dtypes; (void)out_dtype; (void)d_out_all;
  if (nranks != NRANK || (unsigned)rank >= (unsigned)NRANK || n_in != 2 || ws_bytes < WS_NEED ||
      in_sizes[0] != 4096 * 512 || in_sizes[1] != KK * NN || out_size != 512 * NN) {
    fprintf(stderr, "kernel_launch: unexpected problem shape\n");
    return;
  }
  static bool init = false;
  if (!init) {
    HIP_CHECK(hipFuncSetAttribute((const void*)a2a_gemm_q8, hipFuncAttributeMaxDynamicSharedMemorySize, LDS_BYTES));
    init = true;
  }
  Params P;
  P.x = (const char*)d_in[0];
  P.w = (const char*)d_in[1];
  P.out = (signed char*)d_out;
  for (int p = 0; p < NRANK; ++p) P.ws[p] = (char*)d_ws_all[p];
  P.rank = rank;
  hipLaunchKernelGGL(a2a_gemm_q8, dim3(NWG), dim3(NTHREADS), LDS_BYTES, stream, P);
  HIP_CHECK(hipGetLastError());
}
```

```cpp
#include <hip/hip_runtime.h>
#include <rccl/rccl.h>
#include <cstdio>
#include <type_traits>

#define HIP_CHECK(expr) do { hipError_t _e = (expr); if (_e != hipSuccess) { \
    fprintf(stderr, "HIP error %s at %s:%d\n", hipGetErrorString(_e), __FILE__, __LINE__); \
    return; } } while (0)

#define LAS __attribute__((address_space(3)))
#define CAS __attribute__((address_space(4)))
typedef short bf16x8 __attribute__((ext_vector_type(8)));
typedef short bf16x4 __attribute__((ext_vector_type(4)));
typedef float f32x4 __attribute__((ext_vector_type(4)));
typedef unsigned u32x4 __attribute__((ext_vector_type(4)));
typedef unsigned long long u64;

namespace {

constexpr int NRANK = 8;
constexpr int NN = 16384, KK = 4096;
constexpr int NT = KK / 64;
constexpr int A_ST = 32768, B_ST = 16384, NA = 3, NB = 4;
constexpr int B_OFF = 0, A_OFF = NB * B_ST, LDS_BYTES = NA * A_ST + NB * B_ST;
constexpr int NTHREADS = 768, NWG = 256;

constexpr size_t WS_SEQ = 0, WS_TMO = 128;
constexpr size_t WS_FLAGS = 4096;
constexpr size_t WS_LGRAN = 8192;
constexpr size_t WS_GGRAN = 16384;
constexpr int NREP = 32;
constexpr size_t WS_STAGE = 1u << 20;
constexpr size_t WS_NEED = WS_STAGE + (size_t)56 * 65536;

#define SWAPK 0
#define PFA 0
#define PFA_FIRST 1
#define PFA_LAG 12
#define PFA_SLEEP 4
constexpr u64 DEADLINE_TICKS = 100000000ull;
#define PACE 10

struct Params {
  const char* x;
  const char* w;
  signed char* out;
  char* ws[NRANK];
  int rank;
};
#define STAMP3(i) do { } while (0)
#define STAMP2(i) do { } while (0)
#define STAMPC2(i) do { } while (0)
#define STAMPC(i) do { } while (0)
#define STAMP(i) do { } while (0)

#define WAIT_V(n) asm volatile("s_waitcnt vmcnt(" #n ")" ::: "memory")
#define WAIT_L(n) asm volatile("s_waitcnt lgkmcnt(" #n ")" ::: "memory")
#define BAR __builtin_amdgcn_s_barrier()
#define SCHED __builtin_amdgcn_sched_barrier(0)
#define IC(x) std::integral_constant<int, (x)>{}

#define DEF_DMA4(NAME, POL) \
__device__ __forceinline__ void NAME(unsigned lds_addr, const char* gbase, unsigned v0, unsigned v1, unsigned v2, unsigned v3) { \
  asm volatile("s_mov_b32 m0, %0\n\ts_nop 0\n\t" \
               "global_load_lds_dwordx4 %2, %1" POL "\n\t" \
               "global_load_lds_dwordx4 %3, %1 offset:1024" POL "\n\t" \
               "global_load_lds_dwordx4 %4, %1 offset:2048" POL "\n\t" \
               "global_load_lds_dwordx4 %5, %1 offset:3072" POL \
               :: "s"(lds_addr), "s"(gbase), "v"(v0), "v"(v1), "v"(v2), "v"(v3) : "memory"); \
}
#define WPOL ""
DEF_DMA4(dma4, "")
DEF_DMA4(dma4_w, WPOL)
DEF_DMA4(dma4_sys, " sc0 sc1")

__device__ __forceinline__ unsigned sload_u32(const char* p) { return *(const CAS unsigned*)p; }
__device__ __forceinline__ bool vm_busy() { unsigned r; asm volatile("s_getreg_b32 %0, hwreg(HW_REG_IB_STS)" : "=s"(r) :: "memory"); return (r & 0x00c0000fu) != 0u; }
__device__ __forceinline__ u64 now() { return __builtin_amdgcn_s_memrealtime(); }
__device__ __forceinline__ bool timed_out(unsigned* tmo, unsigned seq) {
  return (unsigned)__builtin_amdgcn_readfirstlane(__hip_atomic_load(tmo, __ATOMIC_RELAXED, __HIP_MEMORY_SCOPE_AGENT)) == seq;
}
__device__ __forceinline__ void set_timeout(unsigned* tmo, unsigned seq) { __hip_atomic_store(tmo, seq, __ATOMIC_RELAXED, __HIP_MEMORY_SCOPE_AGENT); }

__device__ __forceinline__ unsigned wave_max_u32(unsigned v) {
  v = max(v, (unsigned)__builtin_amdgcn_update_dpp(0, (int)v, 0x111, 0xf, 0xf, true));
  v = max(v, (unsigned)__builtin_amdgcn_update_dpp(0, (int)v, 0x112, 0xf, 0xf, true));
  v = max(v, (unsigned)__builtin_amdgcn_update_dpp(0, (int)v, 0x114, 0xf, 0xf, true));
  v = max(v, (unsigned)__builtin_amdgcn_update_dpp(0, (int)v, 0x118, 0xf, 0xf, true));
  v = max(v, (unsigned)__builtin_amdgcn_update_dpp(0, (int)v, 0x142, 0xa, 0xf, true));
  v = max(v, (unsigned)__builtin_amdgcn_update_dpp(0, (int)v, 0x143, 0xc, 0xf, true));
  return (unsigned)__builtin_amdgcn_readlane((int)v, 63);
}

__device__ __forceinline__ int step_k(int t, int rank) {
  if (t < 8) return rank * 512 + t * 64;
  const int u = t - 8, j = (u * 37) >> 8, i = u - j * 7;
  return ((rank + 1 + i) & 7) * 512 + j * 64;
}

__global__ __launch_bounds__(NTHREADS) void a2a_gemm_q8(Params P) {
  extern __shared__ __attribute__((aligned(16))) unsigned char shm[];
  LAS unsigned char* lds = (LAS unsigned char*)shm;
  const unsigned ldsb = (unsigned)(size_t)lds;
  const int tid = threadIdx.x, wid = __builtin_amdgcn_readfirstlane(tid >> 6), lane = tid & 63;
  const int b = blockIdx.x, xcd = b & 7, idx = b >> 3;
  const int ct = xcd * 16 + (idx >> 1), rt = idx & 1;
  const int rank = P.rank;
  const int sw = rt & SWAPK;
  char* const myws = P.ws[rank];
  unsigned* const tmo = (unsigned*)(myws + WS_TMO);

  if (wid >= 8) {
    if (wid < 10) {
      const int a = wid - 8;
      unsigned vS[2], vL[4];
#pragma unroll
      for (int h = 0; h < 2; ++h) { const int f = (4 * h + (lane >> 4)) & 7; vS[h] = (unsigned)((lane >> 3) * 128 + (((lane & 7) ^ f) * 16)); }
#pragma unroll
      for (int i = 0; i < 4; ++i) { const int f = (4 * i + (lane >> 4)) & 7; vL[i] = (unsigned)((lane >> 3) * 1024 + (((lane & 7) ^ f) * 16) + i * (8192 - 1024)); }
      const char* xb = P.x + (size_t)(rank * 512 + rt * 256 + a * 128) * 1024;
      const char* sbp = myws + WS_STAGE + (size_t)rt * 32768 + a * 16384;
      const char* flags = myws + WS_FLAGS;
      const unsigned ldst = ldsb + A_OFF + a * 16384;
      const unsigned foff = (unsigned)(lane & 7) * 16;
      auto issue_own = [&](unsigned l, const char* g) {
        dma4(l, g, vL[0], vL[1], vL[2], vL[3]); dma4(l + 4096, g + 32 * 1024, vL[0], vL[1], vL[2], vL[3]);
        dma4(l + 8192, g + 64 * 1024, vL[0], vL[1], vL[2], vL[3]); dma4(l + 12288, g + 96 * 1024, vL[0], vL[1], vL[2], vL[3]);
      };
      auto issue_peer = [&](unsigned l, const char* g) {
        dma4_sys(l, g, vS[0], vS[1], vS[0], vS[1]); dma4_sys(l + 4096, g + 4096, vS[0], vS[1], vS[0], vS[1]);
        dma4_sys(l + 8192, g + 8192, vS[0], vS[1], vS[0], vS[1]); dma4_sys(l + 12288, g + 12288, vS[0], vS[1], vS[0], vS[1]);
      };
      issue_own(ldst, xb + sw * 128); issue_own(ldst + A_ST, xb + (1 ^ sw) * 128);
      const unsigned seq = sload_u32(myws + WS_SEQ) + 1u;
      u64 ready = 0; bool dead = false;
      unsigned l_nx = ldst + 2 * A_ST; const char* g_nx = xb + (2 ^ sw) * 128; bool own_nx = true, poll_nx = false; int j_nx = 0;
      WAIT_V(16); BAR;
#define AW0 do { } while (0)
#define AW1 do { } while (0)
      for (int t = 0; t < NT; ++t) {
        const bool live = t + 2 < NT && !dead;
        if (live && poll_nx) {
          u32x4 f;
          asm volatile("global_load_dwordx4 %0, %1, %2 sc0 sc1" : "=&v"(f) : "v"(foff), "s"(flags + j_nx * 128) : "memory");
          if (own_nx) issue_own(l_nx, g_nx); else issue_peer(l_nx, g_nx);
          BAR;
          AW0; WAIT_V(16); AW1;
          asm volatile("" : "+v"(f));
          const bool ok = f[0] >= seq && f[1] >= seq && f[2] >= seq && f[3] >= seq;
          ready |= (u64)((unsigned)__builtin_amdgcn_ballot_w64(ok) & 0xffu) << (8 * j_nx);
        } else if (live) {
          if (own_nx) issue_own(l_nx, g_nx); else issue_peer(l_nx, g_nx);
          BAR;
          AW0; WAIT_V(16); AW1;
        } else {
          BAR;
          WAIT_V(0);
        }
        const int s = t + 3;
        if (s < NT && !dead) {
          const int k = s ^ sw;
          l_nx = l_nx == ldst + 2 * A_ST ? ldst : l_nx + A_ST;
          own_nx = k < 8;
          if (own_nx) g_nx = xb + k * 128;
          else {
            const int u = k - 8, j = (u * 37) >> 8, src = (rank + 1 + (u - j * 7)) & 7;
            g_nx = sbp + (size_t)u * 65536;
            if (!((ready >> (8 * j + src)) & 1ull)) {
              const u64 t0 = now();
              if (a == 0) STAMP3(64 + (s - 8));
              for (;;) {
                u32x4 f;
                asm volatile("global_load_dwordx4 %0, %1, %2 sc0 sc1\n\ts_waitcnt vmcnt(0)" : "=&v"(f) : "v"(foff), "s"(flags + j * 128) : "memory");
                const bool ok = f[0] >= seq && f[1] >= seq && f[2] >= seq && f[3] >= seq;
                ready |= (u64)((unsigned)__builtin_amdgcn_ballot_w64(ok) & 0xffu) << (8 * j);
                if ((ready >> (8 * j + src)) & 1ull) break;
                if (timed_out(tmo, seq) || now() - t0 > DEADLINE_TICKS) { set_timeout(tmo, seq); dead = true; break; }
                __builtin_amdgcn_s_sleep(1);
              }
            }
          }
          poll_nx = false;
          if (s + 1 >= 8 && s + 1 < NT) {
            const int u = ((s + 1) ^ sw) - 8; j_nx = (u * 37) >> 8; const int src = (rank + 1 + (u - j_nx * 7)) & 7;
            poll_nx = !((ready >> (8 * j_nx + src)) & 1ull);
          }
        }
        BAR;
      }
      BAR;
    } else if (wid == 10) {
      unsigned vB[4];
#pragma unroll
      for (int i = 0; i < 4; ++i) { const int q = lane >> 4, s = (q << 1) | ((i >> 1) & 1), u = ((lane & 15) >> 1) ^ s;
        vB[i] = (unsigned)(q * (NN * 2) + (2 * u + (lane & 1)) * 16 + i * (4 * NN * 2 - 1024)); }
      const char* wb = P.w + (size_t)ct * 256;
      const unsigned ldst = ldsb + B_OFF;
      auto issue = [&](int t) {
        const char* g = wb + (size_t)step_k(t ^ sw, rank) * (NN * 2);
        const unsigned l = ldst + (t % NB) * B_ST;
#pragma unroll
        for (int q = 0; q < 4; ++q) dma4_w(l + q * 4096, g + (size_t)q * 16 * NN * 2, vB[0], vB[1], vB[2], vB[3]);
      };
      issue(0); issue(1); issue(2);
      WAIT_V(32); BAR;
      for (int t = 0; t < NT - 3; ++t) {
        BAR;
        issue(t + 3);
        WAIT_V(32);
        BAR;
      }
      BAR; WAIT_V(16); BAR;
      BAR; WAIT_V(0); BAR;
      BAR; BAR;
      BAR;
    } else {
      const bool has_push = b < 224;
      const int j = has_push ? b / 28 : 8, r = b - j * 28, i = (r >> 2) & 7, q = r & 3;
      const int dst = (rank + 1 + i) & 7;
      u32x4 d[16];
      if (has_push) {
        const char* src = P.x + (size_t)(dst * 512 + q * 128) * 1024 + j * 128;
        const unsigned voff = (unsigned)((lane >> 3) * 1024 + (lane & 7) * 16);
#pragma unroll
        for (int k = 0; k < 16; ++k)
          asm volatile("global_load_dwordx4 %0, %1, %2" : "=&v"(d[k]) : "v"(voff), "s"(src + k * 8192) : "memory");
      }
      const unsigned seq = sload_u32(myws + WS_SEQ) + 1u;
      const u64 t0 = now();
      int s = 0;
      bool pushed = !has_push;
      auto try_push = [&]() {
        if (pushed || s < PACE * j) return;
        char* dws = P.ws[dst];
        char* dbase = dws + WS_STAGE + (size_t)(7 * j + (6 - i)) * 65536 + q * 16384;
        STAMP3(32);
        const unsigned soff = (unsigned)lane * 16;
#pragma unroll
        for (int k = 0; k < 16; ++k)
          asm volatile("global_store_dwordx4 %0, %1, %2 sc0 sc1\n\ts_nop 1" :: "v"(soff), "v"(d[k]), "s"(dbase + k * 1024) : "memory");
        while (vm_busy()) { BAR; ++s; }
        WAIT_V(0);
        STAMP3(34);
        if (lane == 0)
          __hip_atomic_store((unsigned*)(dws + WS_FLAGS + j * 128 + rank * 16 + q * 4), seq, __ATOMIC_RELAXED, __HIP_MEMORY_SCOPE_SYSTEM);
        pushed = true;
        while (vm_busy()) { BAR; ++s; }
      };
      while (vm_busy()) { BAR; ++s; }
      WAIT_V(0);
      try_push();
      while (!pushed) { BAR; ++s; try_push(); }
    }
    return;
  }

  const int grp = wid >> 2, wm = wid & 3, wn = grp;
  const int fr = lane & 15, fq = lane >> 4;
  unsigned aoff[2];
#pragma unroll
  for (int kh = 0; kh < 2; ++kh) aoff[kh] = ldsb + (unsigned)(A_OFF + (wm * 64 + fr) * 128 + (((4 * kh + fq) ^ ((fr >> 1) & 7)) * 16));
  unsigned boff[4];
  { const int q = (lane & 15) >> 2, p = lane & 3; const int s = (q << 1) | (fq & 1);
#pragma unroll
    for (int n = 0; n < 4; ++n) { const int u = (wn * 4 + n) ^ s; boff[n] = ldsb + (unsigned)(B_OFF + (8 * fq + q) * 256 + u * 32 + p * 8); } }
  unsigned aoffH[2] = {aoff[0] + 65536u, aoff[1] + 65536u};
  asm volatile("" : "+v"(aoffH[0]), "+v"(aoffH[1]));

  f32x4 acc[4][4];
#pragma unroll
  for (int m = 0; m < 4; ++m)
#pragma unroll
    for (int n = 0; n < 4; ++n) acc[m][n] = (f32x4){0.f, 0.f, 0.f, 0.f};
  bf16x8 af[4][2], bfr[4][2];
  const unsigned seq = sload_u32(myws + WS_SEQ) + 1u;

  if (wid == 0) STAMP(0);
  BAR;
  if (grp == 1) BAR;
  if (wid == 0) STAMP(1);

#define TRD(dst, addr, off) asm volatile("ds_read_b64_tr_b16 %0, %1 offset:%2" : "=&v"(dst) : "v"(addr), "i"(off) : "memory")
#define ARD(dst, addr, off) asm volatile("ds_read_b128 %0, %1 offset:%2" : "=&v"(dst) : "v"(addr), "i"(off) : "memory")
  auto step = [&](auto SAc, auto SBc) {
    constexpr int SA = decltype(SAc)::value, SB = decltype(SBc)::value;
    bf16x4 lo[4][2], hi[4][2];
#pragma unroll
    for (int n = 0; n < 4; ++n) {
      TRD(lo[n][0], boff[n], SB * B_ST); TRD(hi[n][0], boff[n], SB * B_ST + 1024);
      TRD(lo[n][1], boff[n], SB * B_ST + 8192); TRD(hi[n][1], boff[n], SB * B_ST + 9216);
    }
#pragma unroll
    for (int m = 0; m < 4; ++m) {
      if constexpr (SA < 2) { ARD(af[m][0], aoff[0], SA * A_ST + m * 2048); ARD(af[m][1], aoff[1], SA * A_ST + m * 2048); }
      else { ARD(af[m][0], aoffH[0], m * 2048); ARD(af[m][1], aoffH[1], m * 2048); }
    }
    WAIT_L(0);
    SCHED;
#pragma unroll
    for (int n = 0; n < 4; ++n)
#pragma unroll
      for (int kh = 0; kh < 2; ++kh) bfr[n][kh] = __builtin_shufflevector(lo[n][kh], hi[n][kh], 0, 1, 2, 3, 4, 5, 6, 7);
    BAR;
    __builtin_amdgcn_s_setprio(1);
#pragma unroll
    for (int kh = 0; kh < 2; ++kh)
#pragma unroll
      for (int m = 0; m < 4; ++m)
#pragma unroll
        for (int n = 0; n < 4; ++n) acc[m][n] = __builtin_amdgcn_mfma_f32_16x16x32_bf16(bfr[n][kh], af[m][kh], acc[m][n], 0, 0, 0);
    __builtin_amdgcn_s_setprio(0);
    BAR;
    SCHED;
  };
  for (int it = 0; it < 5; ++it) {
    step(IC(0), IC(0)); step(IC(1), IC(1)); step(IC(2), IC(2)); step(IC(0), IC(3));
    if (wid == 0) { STAMP3(8 + 3 * it); STAMPC2(40 + it); }
    step(IC(1), IC(0)); step(IC(2), IC(1)); step(IC(0), IC(2)); step(IC(1), IC(3));
    if (wid == 0) STAMP3(9 + 3 * it);
    step(IC(2), IC(0)); step(IC(0), IC(1)); step(IC(1), IC(2)); step(IC(2), IC(3));
    if (wid == 0) STAMP3(10 + 3 * it);
  }
  step(IC(0), IC(0)); step(IC(1), IC(1)); step(IC(2), IC(2)); step(IC(0), IC(3));

  if (wid == 0) STAMP(2);
  float mx = 0.f;
#pragma unroll
  for (int m = 0; m < 4; ++m)
#pragma unroll
    for (int n = 0; n < 4; ++n) {
      mx = __builtin_fmaxf(mx, __builtin_fmaxf(__builtin_fabsf(acc[m][n][0]), __builtin_fabsf(acc[m][n][1])));
      mx = __builtin_fmaxf(mx, __builtin_fmaxf(__builtin_fabsf(acc[m][n][2]), __builtin_fabsf(acc[m][n][3])));
    }
  const unsigned wmx = wave_max_u32(__float_as_uint(mx));
  if (grp == 0) BAR;
  LAS unsigned* red = (LAS unsigned*)lds;
  if (lane == 0) red[wid] = wmx;
  WAIT_L(0); BAR;

  if (wid == 0) {
    unsigned v = red[lane & 7];
    v = max(v, (unsigned)__builtin_amdgcn_update_dpp(0, (int)v, 0x111, 0xf, 0xf, true));
    v = max(v, (unsigned)__builtin_amdgcn_update_dpp(0, (int)v, 0x112, 0xf, 0xf, true));
    v = max(v, (unsigned)__builtin_amdgcn_update_dpp(0, (int)v, 0x114, 0xf, 0xf, true));
    const unsigned gmx = (unsigned)__builtin_amdgcn_readlane((int)v, 7);
    u64* lg = (u64*)(myws + WS_LGRAN);
    if (lane == 0) __hip_atomic_store(lg + b, ((u64)seq << 32) | gmx, __ATOMIC_RELAXED, __HIP_MEMORY_SCOPE_AGENT);
    STAMP(3);
    bool dead = false;
    if (b < NRANK) {
      const u64 t0 = now(); unsigned am = 0;
      for (unsigned it = 1;; ++it) {
        bool ok = true; am = 0;
#pragma unroll
        for (int k = 0; k < 4; ++k) { const u64 g = __hip_atomic_load(lg + k * 64 + lane, __ATOMIC_RELAXED, __HIP_MEMORY_SCOPE_AGENT);
          ok = ok && (unsigned)(g >> 32) >= seq; am = max(am, (unsigned)g); }
        if (__builtin_amdgcn_ballot_w64(ok) == ~0ull) break;
        if ((it & 15u) == 0u && (timed_out(tmo, seq) || now() - t0 > DEADLINE_TICKS)) { set_timeout(tmo, seq); dead = true; break; }
      }
      am = wave_max_u32(am);
      if (!dead && lane < NREP)
        __hip_atomic_store((u64*)(P.ws[b] + WS_GGRAN + lane * 128 + rank * 8), ((u64)seq << 32) | am, __ATOMIC_RELAXED, __HIP_MEMORY_SCOPE_SYSTEM);
    }
    STAMP(4);
    const u64* gg = (const u64*)(myws + WS_GGRAN + (b & (NREP - 1)) * 128) + (lane & 7);
    const u64 t0 = now(); unsigned am = 0;
    for (unsigned it = 1; !dead; ++it) {
      const u64 g = __hip_atomic_load(gg, __ATOMIC_RELAXED, __HIP_MEMORY_SCOPE_SYSTEM);
      am = (unsigned)g;
      if (__builtin_amdgcn_ballot_w64((unsigned)(g >> 32) >= seq) == ~0ull) break;
      if ((it & 15u) == 0u && (timed_out(tmo, seq) || now() - t0 > DEADLINE_TICKS)) { set_timeout(tmo, seq); dead = true; break; }
      __builtin_amdgcn_s_sleep(1);
    }
    am = max(am, (unsigned)__builtin_amdgcn_update_dpp(0, (int)am, 0x111, 0xf, 0xf, true));
    am = max(am, (unsigned)__builtin_amdgcn_update_dpp(0, (int)am, 0x112, 0xf, 0xf, true));
    am = max(am, (unsigned)__builtin_amdgcn_update_dpp(0, (int)am, 0x114, 0xf, 0xf, true));
    am = (unsigned)__builtin_amdgcn_readlane((int)am, 7);
    STAMP(5);
    const bool bad = dead || timed_out(tmo, seq);
    if (lane == 0) { red[8] = am; red[9] = bad ? 1u : 0u;
      if (b == 0) __hip_atomic_store((unsigned*)(myws + WS_SEQ), seq, __ATOMIC_RELAXED, __HIP_MEMORY_SCOPE_AGENT); }
  }
  WAIT_L(0); BAR;
  const float amax = __uint_as_float(red[8]);
  const bool bad = red[9] != 0u;
  const float inv = amax > 0.f ? 1.0f / (amax / 127.0f) : 0.f;

  signed char* orow = P.out + (size_t)(rt * 256 + wm * 64 + fr) * NN + ct * 128 + wn * 64 + fq * 16;
#pragma unroll
  for (int m = 0; m < 4; ++m) {
    unsigned d[4];
#pragma unroll
    for (int n = 0; n < 4; ++n) {
      unsigned q[4];
#pragma unroll
      for (int e = 0; e < 4; ++e) q[e] = __float_as_uint(__builtin_fmaf(acc[m][n][e], inv, 12582912.0f));
      const unsigned p01 = __builtin_amdgcn_perm(q[1], q[0], 0x0c0c0400u), p23 = __builtin_amdgcn_perm(q[3], q[2], 0x04000c0cu);
      d[n] = p01 | p23;
    }
    auto r0 = __builtin_amdgcn_permlane32_swap(d[0], d[2], false, false); d[0] = r0[0]; d[2] = r0[1];
    auto r1 = __builtin_amdgcn_permlane32_swap(d[1], d[3], false, false); d[1] = r1[0]; d[3] = r1[1];
    auto r2 = __builtin_amdgcn_permlane16_swap(d[0], d[1], false, false); d[0] = r2[0]; d[1] = r2[1];
    auto r3 = __builtin_amdgcn_permlane16_swap(d[2], d[3], false, false); d[2] = r3[0]; d[3] = r3[1];
    u32x4 o = (u32x4){d[0], d[1], d[2], d[3]};
    if (bad) o = (u32x4){0x80808080u, 0x80808080u, 0x80808080u, 0x80808080u};
    *(u32x4*)(orow + (size_t)m * 16 * NN) = o;
  }
  if (wid == 0) STAMP(6);
}

}

extern "C" void kernel_launch(int rank, int nranks, ncclComm_t comm,
                              hipStream_t stream,
                              void* const* d_in, const int* in_sizes, int n_in,
                              void* d_out, int out_size,
                              const int* in_dtypes, int out_dtype,
                              void* const* d_out_all,
                              void* const* d_ws_all, size_t ws_bytes) {
  (void)comm; (void)in_dtypes; (void)out_dtype; (void)d_out_all;
  if (nranks != NRANK || (unsigned)rank >= (unsigned)NRANK || n_in != 2 || ws_bytes < WS_NEED ||
      in_sizes[0] != 4096 * 512 || in_sizes[1] != KK * NN || out_size != 512 * NN) {
    fprintf(stderr, "kernel_launch: unexpected problem shape\n");
    return;
  }
  static bool init = false;
  if (!init) {
    HIP_CHECK(hipFuncSetAttribute((const void*)a2a_gemm_q8, hipFuncAttributeMaxDynamicSharedMemorySize, LDS_BYTES));
    init = true;
  }
  Params P;
  P.x = (const char*)d_in[0];
  P.w = (const char*)d_in[1];
  P.out = (signed char*)d_out;
  for (int p = 0; p < NRANK; ++p) P.ws[p] = (char*)d_ws_all[p];
  P.rank = rank;
  hipLaunchKernelGGL(a2a_gemm_q8, dim3(NWG), dim3(NTHREADS), LDS_BYTES, stream, P);
  HIP_CHECK(hipGetLastError());
}
```

```cpp
#include <hip/hip_runtime.h>
#include <rccl/rccl.h>
#include <cstdio>
#include <type_traits>

#define HIP_CHECK(expr) do { hipError_t _e = (expr); if (_e != hipSuccess) { \
    fprintf(stderr, "HIP error %s at %s:%d\n", hipGetErrorString(_e), __FILE__, __LINE__); \
    return; } } while (0)

#define LAS __attribute__((address_space(3)))
#define CAS __attribute__((address_space(4)))
typedef short bf16x8 __attribute__((ext_vector_type(8)));
typedef short bf16x4 __attribute__((ext_vector_type(4)));
typedef float f32x4 __attribute__((ext_vector_type(4)));
typedef float f32x2 __attribute__((ext_vector_type(2)));
typedef unsigned u32x4 __attribute__((ext_vector_type(4)));
typedef unsigned long long u64;

namespace {

constexpr int NRANK = 8;
constexpr int NN = 16384, KK = 4096;
constexpr int NT = KK / 64;
constexpr int A_ST = 32768, B_ST = 16384, NA = 3, NB = 4;
constexpr int B_OFF = 0, A_OFF = NB * B_ST, LDS_BYTES = NA * A_ST + NB * B_ST;
constexpr int NTHREADS = 512, NWG = 256;

constexpr size_t WS_SEQ = 0, WS_TMO = 128;
constexpr size_t WS_FLAG0 = 4096;
constexpr size_t WS_FLAG1 = 5120;
constexpr size_t WS_LGRAN = 8192;
constexpr size_t WS_GGRAN = 16384;
constexpr int NREP = 32;
constexpr size_t WS_STAGE = 1u << 20;
constexpr size_t WS_NEED = WS_STAGE + (size_t)56 * 65536;

constexpr u64 DEADLINE_TICKS = 100000000ull;
constexpr int PSTART = 3, PACE = 4, PBATCH = 8;
constexpr int WSLEEP = 6;

struct Params {
  const char* x;
  const char* w;
  signed char* out;
  char* ws[NRANK];
  int rank;
};

#define WAIT_V(n) asm volatile("s_waitcnt vmcnt(" #n ")" ::: "memory")
#define WAIT_L(n) asm volatile("s_waitcnt lgkmcnt(" #n ")" ::: "memory")
#define BAR do { asm volatile("" ::: "memory"); __builtin_amdgcn_s_barrier(); asm volatile("" ::: "memory"); } while (0)
#define SCHED __builtin_amdgcn_sched_barrier(0)
#define IC(x) std::integral_constant<int, (x)>{}
#define INL __attribute__((always_inline))

#define DEF_DMA4(NAME, POL) \
__device__ __forceinline__ void NAME(unsigned lds_addr, const char* gbase, unsigned v0, unsigned v1, unsigned v2, unsigned v3) { \
  asm volatile("s_mov_b32 m0, %0\n\ts_nop 0\n\t" \
               "global_load_lds_dwordx4 %2, %1" POL "\n\t" \
               "global_load_lds_dwordx4 %3, %1 offset:1024" POL "\n\t" \
               "global_load_lds_dwordx4 %4, %1 offset:2048" POL "\n\t" \
               "global_load_lds_dwordx4 %5, %1 offset:3072" POL \
               :: "s"(lds_addr), "s"(gbase), "v"(v0), "v"(v1), "v"(v2), "v"(v3) : "memory"); \
}
DEF_DMA4(dma4, "")
DEF_DMA4(dma4_w, " nt")
DEF_DMA4(dma4_sys, " sc0 sc1")

__device__ __forceinline__ unsigned sload_u32(const char* p) { return *(const CAS unsigned*)p; }
__device__ __forceinline__ unsigned vm_count() { unsigned r; asm volatile("s_getreg_b32 %0, hwreg(HW_REG_IB_STS)" : "=s"(r) :: "memory"); return (r & 0xfu) | ((r >> 18) & 0x30u); }
__device__ __forceinline__ bool vm_busy() { return vm_count() != 0u; }
__device__ __forceinline__ u64 now() { return __builtin_amdgcn_s_memrealtime(); }
__device__ __forceinline__ bool timed_out(unsigned* tmo, unsigned seq) {
  return (unsigned)__builtin_amdgcn_readfirstlane(__hip_atomic_load(tmo, __ATOMIC_RELAXED, __HIP_MEMORY_SCOPE_AGENT)) == seq;
}
__device__ __forceinline__ void set_timeout(unsigned* tmo, unsigned seq) { __hip_atomic_store(tmo, seq, __ATOMIC_RELAXED, __HIP_MEMORY_SCOPE_AGENT); }

__device__ __forceinline__ unsigned wave_max_u32(unsigned v) {
  v = max(v, (unsigned)__builtin_amdgcn_update_dpp(0, (int)v, 0x111, 0xf, 0xf, true));
  v = max(v, (unsigned)__builtin_amdgcn_update_dpp(0, (int)v, 0x112, 0xf, 0xf, true));
  v = max(v, (unsigned)__builtin_amdgcn_update_dpp(0, (int)v, 0x114, 0xf, 0xf, true));
  v = max(v, (unsigned)__builtin_amdgcn_update_dpp(0, (int)v, 0x118, 0xf, 0xf, true));
  v = max(v, (unsigned)__builtin_amdgcn_update_dpp(0, (int)v, 0x142, 0xa, 0xf, true));
  v = max(v, (unsigned)__builtin_amdgcn_update_dpp(0, (int)v, 0x143, 0xc, 0xf, true));
  return (unsigned)__builtin_amdgcn_readlane((int)v, 63);
}

__device__ __forceinline__ int step_k(int t, int rank) {
  if (t < 8) return rank * 512 + t * 64;
  const int u = t - 8, j = (u * 37) >> 8, i = u - j * 7;
  return ((rank + 1 + i) & 7) * 512 + j * 64;
}

__global__ __launch_bounds__(NTHREADS) void a2a_gemm_q8(Params P) {
  extern __shared__ __attribute__((aligned(16))) unsigned char shm[];
  LAS unsigned char* lds = (LAS unsigned char*)shm;
  const unsigned ldsb = (unsigned)(size_t)lds;
  const int tid = threadIdx.x, wid = __builtin_amdgcn_readfirstlane(tid >> 6), lane = tid & 63;
  const int b = blockIdx.x, xcd = b & 7, idx = b >> 3;
  const int ct = xcd * 16 + (idx >> 1), rt = idx & 1;
  const int rank = P.rank;
  char* const myws = P.ws[rank];
  unsigned* const tmo = (unsigned*)(myws + WS_TMO);

  unsigned vL[4], vB[4];
#pragma unroll
  for (int i = 0; i < 4; ++i) { const int f = (4 * i + (lane >> 4)) & 7; vL[i] = (unsigned)((lane >> 3) * 1024 + (((lane & 7) ^ f) * 16) + i * (8192 - 1024)); }
#pragma unroll
  for (int i = 0; i < 4; ++i) { const int q = lane >> 4, s = (q << 1) | ((i >> 1) & 1), u = ((lane & 15) >> 1) ^ s;
    vB[i] = (unsigned)(q * (NN * 2) + (2 * u + (lane & 1)) * 16 + i * (4 * NN * 2 - 1024)); }
  const char* const wb = P.w + (size_t)ct * 256;
  if (wid < 4) {
    const char* g = P.x + (size_t)(rank * 512 + rt * 256 + wid * 64) * 1024; const unsigned l = ldsb + A_OFF + wid * 8192;
    dma4(l, g, vL[0], vL[1], vL[2], vL[3]); dma4(l + 4096, g + 32 * 1024, vL[0], vL[1], vL[2], vL[3]);
  } else if (wid > 4) {
    const int gq = wid - 5;
    const char* g = wb + (size_t)(rank * 512 + gq * 16) * (NN * 2); const unsigned l = ldsb + B_OFF + gq * 4096;
    dma4_w(l, g, vB[0], vB[1], vB[2], vB[3]);
    if (wid == 7) dma4_w(l + 4096, g + (size_t)16 * NN * 2, vB[0], vB[1], vB[2], vB[3]);
  }

  if (__builtin_expect(wid >= 4, 1)) {
    if (wid == 5 || wid == 6) {
      const int a = wid - 5;
      unsigned vS[2];
#pragma unroll
      for (int h = 0; h < 2; ++h) { const int f = (4 * h + (lane >> 4)) & 7; vS[h] = (unsigned)((lane >> 3) * 128 + (((lane & 7) ^ f) * 16)); }
      const char* xb = P.x + (size_t)(rank * 512 + rt * 256 + a * 128) * 1024;
      const char* sbp = myws + WS_STAGE + (size_t)rt * 32768 + a * 16384;
      const unsigned ldst = ldsb + A_OFF + a * 16384;
      const int l0 = lane < 56 ? lane : 55, l1 = lane < 49 ? lane : 48, j1 = (l1 * 37) >> 8;
      const unsigned foff0 = (unsigned)(WS_FLAG0 + ((rank + 1 + (l0 >> 3)) & 7) * 128 + (l0 & 7) * 16);
      const unsigned foff1 = (unsigned)(WS_FLAG1 + (1 + j1) * 128 + ((rank + 1 + (l1 - j1 * 7)) & 7) * 16);
      auto issue_own = [&](unsigned l, const char* g) INL {
        dma4(l, g, vL[0], vL[1], vL[2], vL[3]); dma4(l + 4096, g + 32 * 1024, vL[0], vL[1], vL[2], vL[3]);
        dma4(l + 8192, g + 64 * 1024, vL[0], vL[1], vL[2], vL[3]); dma4(l + 12288, g + 96 * 1024, vL[0], vL[1], vL[2], vL[3]);
      };
      issue_own(ldst + A_ST, xb + 128); issue_own(ldst + 2 * A_ST, xb + 256);
      const unsigned seq = sload_u32(myws + WS_SEQ) + 1u;
      WAIT_V(32); BAR;
      u64 sready = 0xffull;
      bool dead = false;
      int nxt = 3; unsigned l_nx = ldst;
      const char* g_nx = xb + 3 * 128; unsigned gstr = 32 * 1024, gstep = 128;
      unsigned v0 = vL[0], v1 = vL[1], v2 = vL[2], v3 = vL[3];
      auto issue_nx = [&]() INL {
        dma4_sys(l_nx, g_nx, v0, v1, v2, v3); dma4_sys(l_nx + 4096, g_nx + gstr, v0, v1, v2, v3);
        dma4_sys(l_nx + 8192, g_nx + 2 * gstr, v0, v1, v2, v3); dma4_sys(l_nx + 12288, g_nx + 3 * gstr, v0, v1, v2, v3);
      };
      auto advance = [&]() INL {
        ++nxt; l_nx = l_nx == ldst + 2 * A_ST ? ldst : l_nx + A_ST; g_nx += gstep;
        if (nxt == 8) { g_nx = sbp; gstr = 4096; gstep = 65536; v0 = vS[0]; v1 = vS[1]; v2 = vS[0]; v3 = vS[1]; }
      };
      auto note = [&](unsigned fmin, bool first) INL {
        u64 m = __builtin_amdgcn_ballot_w64(fmin >= seq);
        if (first) { m &= m >> 1; m &= m >> 2; m &= m >> 4; u64 r = 0;
#pragma unroll
          for (int i = 0; i < 7; ++i) r |= ((m >> (8 * i)) & 1ull) << i;
          sready |= r << 8; }
        else sready |= (m & ((1ull << 49) - 1)) << 15;
      };
#define FMIN(dst, f, N) asm volatile("s_waitcnt vmcnt(" #N ")\n\tv_min3_u32 %0, %1, %2, %3\n\tv_min_u32 %0, %0, %4" : "=&v"(dst) : "v"(f[0]), "v"(f[1]), "v"(f[2]), "v"(f[3]) : "memory")
      auto need = [&]() INL {
        const u64 t0 = now();
        while (!((sready >> nxt) & 1ull) && !dead) {
          u32x4 f; unsigned fm;
          asm volatile("global_load_dwordx4 %0, %1, %2 sc0 sc1" : "=&v"(f) : "v"(nxt < 15 ? foff0 : foff1), "s"(myws) : "memory");
          FMIN(fm, f, 0);
          note(fm, nxt < 15);
          if ((sready >> nxt) & 1ull) break;
          if (timed_out(tmo, seq) || now() - t0 > DEADLINE_TICKS) { set_timeout(tmo, seq); dead = true; }
          __builtin_amdgcn_s_sleep(1);
        }
      };
      bool plain = true;
      WAIT_V(16); BAR;
      int t = 0;
      for (; t < NT - 6; ++t) {
        if (plain) {
          issue_nx();
          advance();
          WAIT_V(16);
        } else {
          if (nxt == t + 2) { need(); if (!dead) issue_nx(); advance(); }
          bool go = ((sready >> nxt) & 1ull) && !dead;
          if (go) { issue_nx(); advance(); }
          const bool first = nxt + __builtin_ctzll(~(sready >> nxt)) < 15;
          u32x4 f; unsigned fm;
          asm volatile("global_load_dwordx4 %0, %1, %2 sc0 sc1" : "=&v"(f) : "v"(first ? foff0 : foff1), "s"(myws) : "memory");
          if (go) WAIT_V(17); else WAIT_V(1);
          BAR;
          ++t;
          if (nxt == t + 2) {
            if (!((sready >> nxt) & 1ull)) { FMIN(fm, f, 0); note(fm, first); need(); }
            if (!dead) issue_nx();
            advance();
          }
          go = ((sready >> nxt) & 1ull) && !dead;
          if (go) { issue_nx(); advance(); FMIN(fm, f, 16); } else FMIN(fm, f, 0);
          note(fm, first);
        }
        plain = nxt == t + 4 && ((sready >> nxt) & 7ull) == 7ull && !dead;
        BAR;
      }
      for (; t < NT - 1; ++t) {
        bool extra = false;
        while (nxt < NT && nxt <= t + 3) {
          need();
          if (!dead) issue_nx();
          extra = nxt == t + 3;
          advance();
        }
        if (extra) WAIT_V(16); else WAIT_V(0);
        BAR;
      }
    } else if (wid == 7) {
      const unsigned ldst = ldsb + B_OFF;
      auto issue = [&](const char* g, int t) INL {
        const unsigned l = ldst + (t % NB) * B_ST;
#pragma unroll
        for (int q = 0; q < 4; ++q) dma4_w(l + q * 4096, g + (size_t)q * 16 * NN * 2, vB[0], vB[1], vB[2], vB[3]);
      };
      const unsigned vrow = (unsigned)step_k(lane, rank) * (NN * 2);
      auto src_of = [&](int t) INL { return wb + (size_t)(unsigned)__builtin_amdgcn_readlane((int)vrow, t); };
      issue(src_of(1), 1); issue(src_of(2), 2);
      WAIT_V(32); BAR;
      issue(src_of(3), 3);
      WAIT_V(32); BAR;
      for (int t = 0; t < NT - 1; ++t) {
        if (t + 4 < NT) {
          const char* g = src_of(t + 4);
          __builtin_amdgcn_s_sleep(WSLEEP);
          issue(g, t + 4);
          WAIT_V(32);
        } else if (t + 3 < NT) WAIT_V(16); else WAIT_V(0);
        BAR;
      }
    } else {
      const unsigned voff = (unsigned)((lane >> 3) * 1024 + (lane & 7) * 16), soff = (unsigned)lane * 16;
      const bool first = b < 224;
      const int i0 = first ? b >> 5 : 0, p0 = b & 31, dst0 = (rank + 1 + i0) & 7;
      const int cls = ((idx >> 1) + (int)((size_t)P.w >> 8)) & 3;
      const int piece = cls != 1 ? (idx >> 3) * 48 + ((cls - 2) & 3) * 16 + (idx & 1) * 8 + xcd : (idx < 8 && !(idx & 1) && xcd < 4 ? 192 + xcd : 196);
      const bool more = piece < 196;
      const int j = more ? 1 + piece / 28 : 1, r = more ? piece % 28 : 0;
      const int i = r >> 2, q = r & 3, dst = (rank + 1 + i) & 7;
      u32x4 e2[2], d[16];
      if (first) { const char* src = P.x + (size_t)(dst0 * 512 + p0 * 16) * 1024;
#pragma unroll
        for (int k = 0; k < 2; ++k) asm volatile("global_load_dwordx4 %0, %1, %2" : "=&v"(e2[k]) : "v"(voff), "s"(src + k * 8192) : "memory"); }
      const unsigned seq = sload_u32(myws + WS_SEQ) + 1u;
      int s = 0;
      auto wait_left = [&](unsigned n) INL {
        for (;;) {
          const u64 c0 = __builtin_amdgcn_s_memtime();
          do { if (vm_count() <= n) return; __builtin_amdgcn_s_sleep(1); } while (__builtin_amdgcn_s_memtime() - c0 < 600);
          BAR; ++s;
        }
      };
      auto wait_idle = [&]() INL { wait_left(0); };
      wait_idle();
      WAIT_V(0);
      if (first) {
        char* dws = P.ws[dst0];
        char* dbase = dws + WS_STAGE + (size_t)(6 - i0) * 65536 + p0 * 2048;
#pragma unroll
        for (int k = 0; k < 2; ++k) asm volatile("global_store_dwordx4 %0, %1, %2 sc0 sc1\n\ts_nop 1" :: "v"(soff), "v"(e2[k]), "s"(dbase + k * 1024) : "memory");
        wait_idle();
        WAIT_V(0);
        if (lane == 0) __hip_atomic_store((unsigned*)(dws + WS_FLAG0 + rank * 128 + p0 * 4), seq, __ATOMIC_RELAXED, __HIP_MEMORY_SCOPE_SYSTEM);
      }
      if (more) { const char* src = P.x + (size_t)(dst * 512 + q * 128) * 1024 + j * 128;
#pragma unroll
        for (int k = 0; k < 16; ++k) asm volatile("global_load_dwordx4 %0, %1, %2" : "=&v"(d[k]) : "v"(voff), "s"(src + k * 8192) : "memory"); }
      if (more) {
        wait_idle();
        WAIT_V(0);
        char* dws = P.ws[dst];
        char* dbase = dws + WS_STAGE + (size_t)(7 * j + (6 - i)) * 65536 + q * 16384;
#pragma unroll
        for (int g = 0; g < 16 / PBATCH; ++g) {
          while (s < PSTART + PACE * (j - 1) + g) { BAR; ++s; }
#pragma unroll
          for (int k = PBATCH * g; k < PBATCH * g + PBATCH; ++k) asm volatile("global_store_dwordx4 %0, %1, %2 sc0 sc1\n\ts_nop 1" :: "v"(soff), "v"(d[k]), "s"(dbase + k * 1024) : "memory");
        }
        wait_idle();
        WAIT_V(0);
        if (lane == 0) __hip_atomic_store((unsigned*)(dws + WS_FLAG1 + j * 128 + rank * 16 + q * 4), seq, __ATOMIC_RELAXED, __HIP_MEMORY_SCOPE_SYSTEM);
      }
    }
    return;
  }

  const int wm = wid >> 1, wn = wid & 1;
  const int fr = lane & 15, fq = lane >> 4;
  unsigned aoff[2];
#pragma unroll
  for (int kh = 0; kh < 2; ++kh) aoff[kh] = ldsb + (unsigned)(A_OFF + (wm * 128 + fr) * 128 + (((4 * kh + fq) ^ ((fr >> 1) & 7)) * 16));
  unsigned boff[4];
  { const int q = (lane & 15) >> 2, p = lane & 3; const int s = (q << 1) | (fq & 1);
#pragma unroll
    for (int n = 0; n < 4; ++n) { const int u = (wn * 4 + n) ^ s; boff[n] = ldsb + (unsigned)(B_OFF + (8 * fq + q) * 256 + u * 32 + p * 8); } }
  unsigned aoffH[2] = {aoff[0] + 65536u, aoff[1] + 65536u};
  asm volatile("" : "+v"(aoffH[0]), "+v"(aoffH[1]));

  f32x4 acc[8][4];
#pragma unroll
  for (int m = 0; m < 8; ++m)
#pragma unroll
    for (int n = 0; n < 4; ++n) acc[m][n] = (f32x4){0.f, 0.f, 0.f, 0.f};
  bf16x8 af[2][8], bfr[2][4];
  const unsigned seq = sload_u32(myws + WS_SEQ) + 1u;

  auto reads = [&](auto SAc, auto SBc, auto KHc) INL {
    constexpr int SA = decltype(SAc)::value, SB = decltype(SBc)::value, KH = decltype(KHc)::value;
#pragma unroll
    for (int n = 0; n < 4; ++n) {
      const bf16x4 lo = __builtin_amdgcn_ds_read_tr16_b64_v4i16((LAS bf16x4*)(size_t)(boff[n] + SB * B_ST + KH * 8192));
      const bf16x4 hi = __builtin_amdgcn_ds_read_tr16_b64_v4i16((LAS bf16x4*)(size_t)(boff[n] + SB * B_ST + KH * 8192 + 1024));
      bfr[KH][n] = __builtin_shufflevector(lo, hi, 0, 1, 2, 3, 4, 5, 6, 7);
    }
#pragma unroll
    for (int m = 0; m < 8; ++m) {
      if constexpr (SA < 2) af[KH][m] = *(const LAS bf16x8*)(size_t)(aoff[KH] + SA * A_ST + m * 2048);
      else af[KH][m] = *(const LAS bf16x8*)(size_t)(aoffH[KH] + m * 2048);
    }
  };
  auto mfmas = [&](auto KHc) INL {
    constexpr int KH = decltype(KHc)::value;
#pragma unroll
    for (int m = 0; m < 8; ++m)
#pragma unroll
      for (int n = 0; n < 4; ++n) acc[m][n] = __builtin_amdgcn_mfma_f32_16x16x32_bf16(bfr[KH][n], af[KH][m], acc[m][n], 0, 0, 0);
  };
  auto interleave = [&]() INL {
#pragma unroll
    for (int i = 0; i < 16; ++i) { __builtin_amdgcn_sched_group_barrier(0x008, 1, 0); __builtin_amdgcn_sched_group_barrier(0x100, 1, 0); }
    __builtin_amdgcn_sched_group_barrier(0x008, 16, 0);
  };
  auto step = [&](auto SAc, auto SBc, auto SA1c, auto SB1c, auto LASTc) INL {
    SCHED;
    reads(SAc, SBc, IC(1));
    mfmas(IC(0));
    interleave();
    SCHED;
    WAIT_L(0);
    BAR;
    SCHED;
    if constexpr (!decltype(LASTc)::value) reads(SA1c, SB1c, IC(0));
    mfmas(IC(1));
    if constexpr (!decltype(LASTc)::value) { interleave(); SCHED; }
  };
  WAIT_V(0);
  BAR;
  reads(IC(0), IC(0), IC(0));
  for (int it = 0; it < 5; ++it) {
    step(IC(0), IC(0), IC(1), IC(1), IC(0)); step(IC(1), IC(1), IC(2), IC(2), IC(0)); step(IC(2), IC(2), IC(0), IC(3), IC(0)); step(IC(0), IC(3), IC(1), IC(0), IC(0));
    step(IC(1), IC(0), IC(2), IC(1), IC(0)); step(IC(2), IC(1), IC(0), IC(2), IC(0)); step(IC(0), IC(2), IC(1), IC(3), IC(0)); step(IC(1), IC(3), IC(2), IC(0), IC(0));
    step(IC(2), IC(0), IC(0), IC(1), IC(0)); step(IC(0), IC(1), IC(1), IC(2), IC(0)); step(IC(1), IC(2), IC(2), IC(3), IC(0)); step(IC(2), IC(3), IC(0), IC(0), IC(0));
  }
  step(IC(0), IC(0), IC(1), IC(1), IC(0)); step(IC(1), IC(1), IC(2), IC(2), IC(0)); step(IC(2), IC(2), IC(0), IC(3), IC(0)); step(IC(0), IC(3), IC(1), IC(0), IC(1));

  SCHED;
  asm volatile("s_nop 15\n\ts_nop 7" ::: "memory");
  SCHED;
  float mx0 = 0.f, mx1 = 0.f;
#pragma unroll
  for (int m = 0; m < 8; ++m)
#pragma unroll
    for (int n = 0; n < 4; ++n) {
      asm("v_max3_f32 %0, |%1|, |%2|, %0" : "+v"(mx0) : "v"(acc[m][n][0]), "v"(acc[m][n][1]));
      asm("v_max3_f32 %0, |%1|, |%2|, %0" : "+v"(mx1) : "v"(acc[m][n][2]), "v"(acc[m][n][3]));
    }
  const float mx = mx0 > mx1 ? mx0 : mx1;
  const unsigned wmx = wave_max_u32(__float_as_uint(mx));
  LAS unsigned* red = (LAS unsigned*)lds;
  if (lane == 0) red[wid] = wmx;
  WAIT_L(0); BAR;

  if (wid == 0) {
    const unsigned gmx = max(max(red[0], red[1]), max(red[2], red[3]));
    u64* lg = (u64*)(myws + WS_LGRAN);
    if (lane == 0) __hip_atomic_store(lg + b, ((u64)seq << 32) | gmx, __ATOMIC_RELAXED, __HIP_MEMORY_SCOPE_AGENT);
    bool dead = false;
    if (b < NRANK) {
      const u64 t0 = now(); unsigned am = 0;
      const unsigned go = (unsigned)lane * 16;
      for (unsigned it = 1;; ++it) {
        u32x4 g0, g1;
        asm volatile("global_load_dwordx4 %0, %2, %3 sc1\n\tglobal_load_dwordx4 %1, %2, %3 offset:1024 sc1\n\ts_waitcnt vmcnt(0)"
                     : "=&v"(g0), "=&v"(g1) : "v"(go), "s"(lg) : "memory");
        const unsigned tmin = min(min(g0[1], g0[3]), min(g1[1], g1[3]));
        am = max(max(g0[0], g0[2]), max(g1[0], g1[2]));
        if (__builtin_amdgcn_ballot_w64(tmin >= seq) == ~0ull) break;
        if ((it & 15u) == 0u && (timed_out(tmo, seq) || now() - t0 > DEADLINE_TICKS)) { set_timeout(tmo, seq); dead = true; break; }
      }
      am = wave_max_u32(am);
      if (!dead && lane < NREP)
        __hip_atomic_store((u64*)(P.ws[b] + WS_GGRAN + lane * 128 + rank * 8), ((u64)seq << 32) | am, __ATOMIC_RELAXED, __HIP_MEMORY_SCOPE_SYSTEM);
    }
    const u64* gg = lane < 8 ? (const u64*)(myws + WS_GGRAN + (b & (NREP - 1)) * 128) + lane : (const u64*)tmo;
    const u64 t0 = now(); unsigned am = 0; bool late = false;
    for (unsigned it = 1; !dead; ++it) {
      const u64 g = __hip_atomic_load(gg, __ATOMIC_RELAXED, __HIP_MEMORY_SCOPE_SYSTEM);
      am = lane < 8 ? (unsigned)g : 0u;
      late = __builtin_amdgcn_ballot_w64(lane >= 8 && (unsigned)g == seq) != 0ull;
      if (late || __builtin_amdgcn_ballot_w64(lane >= 8 || (unsigned)(g >> 32) >= seq) == ~0ull) break;
      if ((it & 15u) == 0u && now() - t0 > DEADLINE_TICKS) { set_timeout(tmo, seq); dead = true; break; }
      __builtin_amdgcn_s_sleep(1);
    }
    am = max(am, (unsigned)__builtin_amdgcn_update_dpp(0, (int)am, 0x111, 0xf, 0xf, true));
    am = max(am, (unsigned)__builtin_amdgcn_update_dpp(0, (int)am, 0x112, 0xf, 0xf, true));
    am = max(am, (unsigned)__builtin_amdgcn_update_dpp(0, (int)am, 0x114, 0xf, 0xf, true));
    am = (unsigned)__builtin_amdgcn_readlane((int)am, 7);
    const bool bad = dead || late;
    if (lane == 0) { red[8] = am; red[9] = bad ? 1u : 0u;
      if (b == 0) __hip_atomic_store((unsigned*)(myws + WS_SEQ), seq, __ATOMIC_RELAXED, __HIP_MEMORY_SCOPE_AGENT); }
  }
  WAIT_L(0); BAR;
  const float amax = __uint_as_float(red[8]);
  const bool bad = red[9] != 0u;
  const float inv = amax > 0.f ? 1.0f / (amax / 127.0f) : 0.f;

  const f32x2 inv2 = {inv, inv}, mag2 = {12582912.0f, 12582912.0f};
  signed char* orow = P.out + (size_t)(rt * 256 + wm * 128 + fr) * NN + ct * 128 + wn * 64 + fq * 16;
#pragma unroll
  for (int m = 0; m < 8; ++m) {
    unsigned d[4];
#pragma unroll
    for (int n = 0; n < 4; ++n) {
      const f32x2 lo2 = __builtin_elementwise_fma((f32x2){acc[m][n][0], acc[m][n][1]}, inv2, mag2), hi2 = __builtin_elementwise_fma((f32x2){acc[m][n][2], acc[m][n][3]}, inv2, mag2);
      const unsigned q[4] = {__float_as_uint(lo2[0]), __float_as_uint(lo2[1]), __float_as_uint(hi2[0]), __float_as_uint(hi2[1])};
      const unsigned p01 = __builtin_amdgcn_perm(q[1], q[0], 0x0c0c0400u), p23 = __builtin_amdgcn_perm(q[3], q[2], 0x04000c0cu);
      d[n] = p01 | p23;
    }
    auto r0 = __builtin_amdgcn_permlane32_swap(d[0], d[2], false, false); d[0] = r0[0]; d[2] = r0[1];
    auto r1 = __builtin_amdgcn_permlane32_swap(d[1], d[3], false, false); d[1] = r1[0]; d[3] = r1[1];
    auto r2 = __builtin_amdgcn_permlane16_swap(d[0], d[1], false, false); d[0] = r2[0]; d[1] = r2[1];
    auto r3 = __builtin_amdgcn_permlane16_swap(d[2], d[3], false, false); d[2] = r3[0]; d[3] = r3[1];
    u32x4 o = (u32x4){d[0], d[1], d[2], d[3]};
    if (bad) o = (u32x4){0x80808080u, 0x80808080u, 0x80808080u, 0x80808080u};
    __builtin_nontemporal_store(o, (u32x4*)(orow + (size_t)m * 16 * NN));
  }
}

}

extern "C" void kernel_launch(int rank, int nranks, ncclComm_t comm,
                              hipStream_t stream,
                              void* const* d_in, const int* in_sizes, int n_in,
                              void* d_out, int out_size,
                              const int* in_dtypes, int out_dtype,
                              void* const* d_out_all,
                              void* const* d_ws_all, size_t ws_bytes) {
  (void)comm; (void)in_dtypes; (void)out_dtype; (void)d_out_all;
  if (nranks != NRANK || (unsigned)rank >= (unsigned)NRANK || n_in != 2 || ws_bytes < WS_NEED ||
      in_sizes[0] != 4096 * 512 || in_sizes[1] != KK * NN || out_size != 512 * NN) {
    fprintf(stderr, "kernel_launch: unexpected problem shape\n");
    return;
  }
  static bool init = false;
  if (!init) {
    HIP_CHECK(hipFuncSetAttribute((const void*)a2a_gemm_q8, hipFuncAttributeMaxDynamicSharedMemorySize, LDS_BYTES));
    int dev = 0, cus = 0, per_cu = 0;
    HIP_CHECK(hipGetDevice(&dev));
    HIP_CHECK(hipDeviceGetAttribute(&cus, hipDeviceAttributeMultiprocessorCount, dev));
    HIP_CHECK(hipOccupancyMaxActiveBlocksPerMultiprocessor(&per_cu, a2a_gemm_q8, NTHREADS, LDS_BYTES));
    if (cus * per_cu < NWG) { fprintf(stderr, "kernel_launch: %d CUs x %d workgroups cannot hold the grid of %d\n", cus, per_cu, NWG); return; }
    init = true;
  }
  Params P;
  P.x = (const char*)d_in[0];
  P.w = (const char*)d_in[1];
  P.out = (signed char*)d_out;
  for (int p = 0; p < NRANK; ++p) P.ws[p] = (char*)d_ws_all[p];
  P.rank = rank;
  hipLaunchKernelGGL(a2a_gemm_q8, dim3(NWG), dim3(NTHREADS), LDS_BYTES, stream, P);
  HIP_CHECK(hipGetLastError());
}
```
